```python
import numpy as np

kernel_name = "gpu_gemm_a2a_m2048_k2048_n2048_f32_relu"


ABSMAX_RTOL = 0.001

COMPUTE_DTYPE = "float32"
EPILOGUE = "relu"
IN_QUANT = None
OUT_QUANT = None
IN_DTYPES = {"x": "float32", "w_mat": "float32"}
WORLD_SIZE = 8

def setup_inputs():

    x = np.random.randn(2048, 2048).astype(np.float32)
    w_mat = (np.random.randn(2048, 2048) / np.sqrt(2048)).astype(np.float32)
    return {"x": x, "w_mat": w_mat}

def _epilogue(y):
    return np.maximum(y, 0.0).astype(np.float32)

def reference(x, w_mat):
    y = (x @ w_mat).astype(np.float32)
    return _epilogue(y)
```

```cpp
#include <hip/hip_runtime.h>
#include <rccl/rccl.h>
#include <cstdio>
#include <cstdint>

#define HIP_CHECK(expr) do { hipError_t _e = (expr); if (_e != hipSuccess) { \
    fprintf(stderr, "HIP error %s at %s:%d\n", hipGetErrorString(_e), __FILE__, __LINE__); \
    return; } } while (0)

typedef _Float16 f16x8 __attribute__((ext_vector_type(8)));
typedef _Float16 f16x4 __attribute__((ext_vector_type(4)));
typedef _Float16 f16x2 __attribute__((ext_vector_type(2)));
typedef float f32x4 __attribute__((ext_vector_type(4)));
typedef float f32x2 __attribute__((ext_vector_type(2)));
typedef unsigned u32x4 __attribute__((ext_vector_type(4)));
typedef unsigned u32x2 __attribute__((ext_vector_type(2)));

constexpr int NRANK = 8;
constexpr int ML = 256;
constexpr int KD = 2048;
constexpr int ND = 2048;
constexpr int NB = 256;
constexpr int NWG = 256;
constexpr int WG_PER_DST = NWG / NRANK;

constexpr unsigned WS_SEQ = 0;
constexpr unsigned WS_FLAG = 4096;
constexpr unsigned WS_TMO = 8192;
constexpr unsigned WS_BYTES_USED = 8192 + 128;
constexpr unsigned long long DEADLINE_TICKS = 50000000ull;

struct Params {
    const float* x;
    const float* w;
    float* out[NRANK];
    unsigned* ws[NRANK];
    int rank;
};

constexpr int AUX_SYS = 17;

__device__ __forceinline__ f16x2 cvt2(float a, float b) {
    f32x2 v = {a, b};
    return __builtin_convertvector(v, f16x2);
}

__global__ void __launch_bounds__(512, 2) gemm_a2a_kernel(Params P) {
    extern __shared__ __attribute__((aligned(16))) char smem[];
    const int tid = threadIdx.x;
    const int lane = tid & 63;
    const int wave = __builtin_amdgcn_readfirstlane(tid >> 6);
    const int b = blockIdx.x;
    const int grp = b & 7, idx = b >> 3;
    const unsigned dst = (unsigned)(grp + P.rank) & 7u;
    const int mt = idx & 7, nt = idx >> 3;
    const int m0 = mt * 32;
    const int n0 = dst * NB + nt * 64;
    const int kw = wave * 256;

    unsigned* myws = P.ws[P.rank];
    unsigned call = __hip_atomic_load(myws + WS_SEQ / 4 + b, __ATOMIC_RELAXED, __HIP_MEMORY_SCOPE_AGENT);

    const __amdgpu_buffer_rsrc_t xr = __builtin_amdgcn_make_buffer_rsrc((void*)P.x, 0, ML * KD * 4, 0x00020000);
    const __amdgpu_buffer_rsrc_t wr = __builtin_amdgcn_make_buffer_rsrc((void*)P.w, 0, KD * ND * 4, 0x00020000);

    const int li = lane & 15, lg = lane >> 4;
    const unsigned xoff = (unsigned)(m0 * KD + kw + lane * 4) * 4u;
    const unsigned woff = (unsigned)((kw + 8 * lg) * ND + n0 + 4 * li) * 4u;

    char* xs = smem + wave * 16384;

    f32x4 xv[32];
    f32x4 wv[8][8];
#define LDX(r) xv[r] = __builtin_bit_cast(f32x4, __builtin_amdgcn_raw_buffer_load_b128(xr, xoff, (r) * KD * 4, 0))
#define LDW(s) _Pragma("unroll") for (int j = 0; j < 8; ++j) \
        wv[s][j] = __builtin_bit_cast(f32x4, __builtin_amdgcn_raw_buffer_load_b128(wr, woff, ((s) * 32 + j) * ND * 4, 0))
#define STX(r) do { f16x4 h; f16x2 a = cvt2(xv[r][0], xv[r][1]), c = cvt2(xv[r][2], xv[r][3]); \
        h[0] = a[0]; h[1] = a[1]; h[2] = c[0]; h[3] = c[1]; \
        *(f16x4*)(xs + (r) * 512 + ((((lane >> 1) ^ ((r) & 15)) << 4) | ((lane & 1) << 3))) = h; } while (0)

#pragma unroll
    for (int r = 0; r < 16; ++r) LDX(r);
    LDW(0);
#pragma unroll
    for (int r = 16; r < 32; ++r) LDX(r);
    LDW(1);
    __builtin_amdgcn_sched_barrier(0);
#pragma unroll
    for (int r = 0; r < 16; ++r) STX(r);
    __builtin_amdgcn_sched_barrier(0);
    LDW(2);
    LDW(3);
    __builtin_amdgcn_sched_barrier(0);
#pragma unroll
    for (int r = 16; r < 32; ++r) STX(r);
    __builtin_amdgcn_sched_barrier(0);
    LDW(4);
    LDW(5);
    __builtin_amdgcn_sched_barrier(0);

    f32x4 acc[2][4];
#pragma unroll
    for (int a = 0; a < 2; ++a)
#pragma unroll
        for (int t = 0; t < 4; ++t) acc[a][t] = (f32x4){0.f, 0.f, 0.f, 0.f};

#define STEP(s) do { \
        f16x8 af[2]; \
        _Pragma("unroll") for (int a = 0; a < 2; ++a) \
            af[a] = *(const f16x8*)(xs + (a * 16 + li) * 512 + ((((s) * 4 + lg) ^ li) << 4)); \
        _Pragma("unroll") for (int t = 0; t < 4; ++t) { \
            f16x8 bf; \
            _Pragma("unroll") for (int j = 0; j < 4; ++j) { \
                f16x2 h = cvt2(wv[s][2 * j][t], wv[s][2 * j + 1][t]); \
                bf[2 * j] = h[0]; bf[2 * j + 1] = h[1]; } \
            _Pragma("unroll") for (int a = 0; a < 2; ++a) \
                acc[a][t] = __builtin_amdgcn_mfma_f32_16x16x32_f16(af[a], bf, acc[a][t], 0, 0, 0); \
        } } while (0)

    STEP(0);
    __builtin_amdgcn_sched_barrier(0);
    LDW(6);
    __builtin_amdgcn_sched_barrier(0);
    STEP(1);
    __builtin_amdgcn_sched_barrier(0);
    LDW(7);
    __builtin_amdgcn_sched_barrier(0);
    STEP(2); STEP(3); STEP(4); STEP(5); STEP(6); STEP(7);

    {
        float* red = (float*)(smem + wave * 16384);
#pragma unroll
        for (int a = 0; a < 2; ++a)
#pragma unroll
            for (int r = 0; r < 4; ++r) {
                f32x4 v = {acc[a][0][r], acc[a][1][r], acc[a][2][r], acc[a][3][r]};
                *(f32x4*)(red + (a * 16 + lg * 4 + r) * 64 + li * 4) = v;
            }
    }
    __syncthreads();
    {
        const int row = wave * 4 + lg;
        f32x4 s = *(const f32x4*)(smem + (row * 64 + li * 4) * 4);
#pragma unroll
        for (int w = 1; w < 8; ++w) s += *(const f32x4*)(smem + w * 16384 + (row * 64 + li * 4) * 4);
        s[0] = fmaxf(s[0], 0.f); s[1] = fmaxf(s[1], 0.f); s[2] = fmaxf(s[2], 0.f); s[3] = fmaxf(s[3], 0.f);
        const __amdgpu_buffer_rsrc_t orr = __builtin_amdgcn_make_buffer_rsrc((void*)P.out[dst], 0, ND * NB * 4, 0x00020000);
        const unsigned ooff = (unsigned)((P.rank * ML + m0 + row) * NB + nt * 64 + li * 4) * 4u;
        __builtin_amdgcn_raw_buffer_store_b128(__builtin_bit_cast(u32x4, s), orr, ooff, 0, AUX_SYS);
    }
    asm volatile("s_waitcnt vmcnt(0)" ::: "memory");
    __syncthreads();
    if (wave == 0) {
        call = __builtin_amdgcn_readfirstlane(call) + 1u;
        if (lane == 0) {
            if (dst != (unsigned)P.rank)
                __hip_atomic_store(P.ws[dst] + WS_FLAG / 4 + P.rank * WG_PER_DST + idx, call,
                                   __ATOMIC_RELAXED, __HIP_MEMORY_SCOPE_SYSTEM);
            __hip_atomic_store(myws + WS_SEQ / 4 + b, call, __ATOMIC_RELAXED, __HIP_MEMORY_SCOPE_AGENT);
        }
        if (dst == (unsigned)P.rank && idx == 0) {
            const __amdgpu_buffer_rsrc_t fr = __builtin_amdgcn_make_buffer_rsrc((void*)(myws + WS_FLAG / 4), 0, NRANK * 128, 0x00020000);
            const bool self = (lane >> 3) == P.rank;
            const unsigned long long tw = __builtin_amdgcn_s_memrealtime();
            bool timed_out = false;
            for (;;) {
                asm volatile("" ::: "memory");
                u32x4 f = __builtin_amdgcn_raw_buffer_load_b128(fr, lane * 16, 0, AUX_SYS);
                bool ok = self || (f[0] >= call && f[1] >= call && f[2] >= call && f[3] >= call);
                if (__all(ok)) break;
                if (__builtin_amdgcn_s_memrealtime() - tw > DEADLINE_TICKS) { timed_out = true; break; }
                __builtin_amdgcn_s_sleep(1);
            }
            if (timed_out && lane == 0) {
                __hip_atomic_store(myws + WS_TMO / 4, call, __ATOMIC_RELAXED, __HIP_MEMORY_SCOPE_AGENT);
                P.out[P.rank][(size_t)(P.rank * ML + m0) * NB + nt * 64] = __builtin_nanf("");
            }
            __builtin_amdgcn_fence(__ATOMIC_ACQUIRE, "");
        }
    }
}

constexpr int LDS_BYTES = 8 * 16384;

extern "C" void kernel_launch(int rank, int nranks, ncclComm_t comm,
                              hipStream_t stream,
                              void* const* d_in, const int* in_sizes, int n_in,
                              void* d_out, int out_size,
                              const int* in_dtypes, int out_dtype,
                              void* const* d_out_all,
                              void* const* d_ws_all, size_t ws_bytes) {
    if (nranks != NRANK || (unsigned)rank >= (unsigned)NRANK || ws_bytes < WS_BYTES_USED) {
        fprintf(stderr, "unexpected configuration\n");
        return;
    }
    Params P;
    P.x = (const float*)d_in[0];
    P.w = (const float*)d_in[1];
    for (int p = 0; p < NRANK; ++p) {
        P.out[p] = (float*)d_out_all[p];
        P.ws[p] = (unsigned*)d_ws_all[p];
    }
    P.rank = rank;
    HIP_CHECK(hipFuncSetAttribute((const void*)gemm_a2a_kernel, hipFuncAttributeMaxDynamicSharedMemorySize, LDS_BYTES));
    hipLaunchKernelGGL(gemm_a2a_kernel, dim3(NWG), dim3(512), LDS_BYTES, stream, P);
    HIP_CHECK(hipGetLastError());
}
```

```cpp
#include <hip/hip_runtime.h>
#include <rccl/rccl.h>
#include <cstdio>
#include <cstdint>

#define HIP_CHECK(expr) do { hipError_t _e = (expr); if (_e != hipSuccess) { \
    fprintf(stderr, "HIP error %s at %s:%d\n", hipGetErrorString(_e), __FILE__, __LINE__); \
    return; } } while (0)

typedef _Float16 f16x8 __attribute__((ext_vector_type(8)));
typedef _Float16 f16x4 __attribute__((ext_vector_type(4)));
typedef _Float16 f16x2 __attribute__((ext_vector_type(2)));
typedef float f32x4 __attribute__((ext_vector_type(4)));
typedef float f32x2 __attribute__((ext_vector_type(2)));
typedef unsigned u32x4 __attribute__((ext_vector_type(4)));
typedef unsigned u32x2 __attribute__((ext_vector_type(2)));

constexpr int NRANK = 8;
constexpr int ML = 256;
constexpr int KD = 2048;
constexpr int ND = 2048;
constexpr int NB = 256;
constexpr int NWG = 256;
constexpr int WG_PER_DST = NWG / NRANK;

constexpr unsigned WS_SEQ = 0;
constexpr unsigned WS_FLAG = 4096;
constexpr unsigned WS_TMO = 8192;
constexpr unsigned WS_BYTES_USED = 8192 + 128;
constexpr unsigned long long DEADLINE_TICKS = 50000000ull;

struct Params {
    const float* x;
    const float* w;
    float* out[NRANK];
    unsigned* ws[NRANK];
    int rank;
};

constexpr int AUX_SYS = 17;

__device__ __forceinline__ unsigned cvt2(float a, float b) {
    f32x2 v = {a, b};
    return __builtin_bit_cast(unsigned, __builtin_convertvector(v, f16x2));
}
#define SB() __builtin_amdgcn_sched_barrier(0)
#define STAMP(i)

constexpr int LDS_XS = 8 * 8192;
constexpr int LDS_RED = 4 * 8 * 2048;
constexpr int LDS_BYTES = LDS_XS + LDS_RED;

__global__ void __launch_bounds__(512, 2) gemm_a2a_kernel(Params P) {
    extern __shared__ __attribute__((aligned(16))) char smem[];
    const int tid = threadIdx.x;
    const int lane = tid & 63;
    const int wave = __builtin_amdgcn_readfirstlane(tid >> 6);
    const int b = blockIdx.x;
    const int grp = b & 7, idx = b >> 3;
    const unsigned dst = (unsigned)(grp + P.rank) & 7u;
    const int mt = idx & 3, nt = idx >> 2;
    const int m0 = mt * 64;
    const int n0 = dst * NB + nt * 32;
    const int kw = wave * 256;
    STAMP(0);

    unsigned* myws = P.ws[P.rank];
    unsigned call = __hip_atomic_load(myws + WS_SEQ / 4 + b, __ATOMIC_RELAXED, __HIP_MEMORY_SCOPE_AGENT);

    const __amdgpu_buffer_rsrc_t xr = __builtin_amdgcn_make_buffer_rsrc((void*)P.x, 0, ML * KD * 4, 0x00020000);
    const __amdgpu_buffer_rsrc_t wr = __builtin_amdgcn_make_buffer_rsrc((void*)P.w, 0, KD * ND * 4, 0x00020000);
    const __amdgpu_buffer_rsrc_t orr = __builtin_amdgcn_make_buffer_rsrc((void*)P.out[dst], 0, ND * NB * 4, 0x00020000);

    const int li = lane & 15, lg = lane >> 4;
    const unsigned xoff = (unsigned)(m0 * KD + kw + lane * 4) * 4u;
    const unsigned woff = (unsigned)((kw + 4 * (lane >> 3)) * ND + n0 + 4 * (lane & 7)) * 4u;
    const int rotw = 2 * mt;
    const int rotx = 2 * nt;

    char* xs = smem + wave * 8192;
    const unsigned xs_w = ((lane >> 1) << 4) | ((lane & 1) << 3);

    f32x4 xa[16], xb[16];
    f32x4 wv[8][4];
    u32x4 wf[8][2];
#define LDX(buf, st) _Pragma("unroll") for (int r = 0; r < 16; ++r) \
        buf[r] = __builtin_bit_cast(f32x4, __builtin_amdgcn_raw_buffer_load_b128(xr, xoff, ((st) * 16 + ((r + rotx) & 15)) * KD * 4, 0))
#define STX(buf) _Pragma("unroll") for (int r = 0; r < 16; ++r) { \
        const int rr = (r + rotx) & 15; \
        u32x2 h = {cvt2(buf[r][0], buf[r][1]), cvt2(buf[r][2], buf[r][3])}; \
        *(u32x2*)(xs + rr * 512 + (xs_w ^ (rr << 4))) = h; }
#define LDW(s) _Pragma("unroll") for (int a = 0; a < 4; ++a) \
        wv[s][a] = __builtin_bit_cast(f32x4, __builtin_amdgcn_raw_buffer_load_b128(wr, woff, ((((s) + rotw) & 7) * 32 + a) * ND * 4, 0))
#define CVW(s) _Pragma("unroll") for (int t = 0; t < 2; ++t) _Pragma("unroll") for (int d = 0; d < 2; ++d) { \
        unsigned p0 = cvt2(wv[s][2 * d][t], wv[s][2 * d + 1][t]); \
        unsigned p2 = cvt2(wv[s][2 * d][2 + t], wv[s][2 * d + 1][2 + t]); \
        wf[s][t][d] = __builtin_amdgcn_update_dpp(p0, p2, 0x128, 0xf, 0xc, false); \
        wf[s][t][2 + d] = __builtin_amdgcn_update_dpp(p2, p0, 0x128, 0xf, 0x3, false); }

    LDX(xa, 0);
#pragma unroll
    for (int s = 0; s < 8; ++s) LDW(s);
    SB();
    STX(xa);
    SB();
    LDX(xa, 1);
    SB();
    CVW(0); CVW(1); CVW(2); CVW(3);
    SB();
    LDX(xb, 2);
    SB();
    CVW(4); CVW(5); CVW(6); CVW(7);
    SB();
    STAMP(1);

#define STAGE(st) do { \
        f32x4 acc0 = {0.f, 0.f, 0.f, 0.f}, acc1 = {0.f, 0.f, 0.f, 0.f}; \
        _Pragma("unroll") for (int s = 0; s < 8; ++s) { \
            const int sr = (s + rotw) & 7; \
            f16x8 af = *(const f16x8*)(xs + li * 512 + (((sr * 4 + lg) ^ li) << 4)); \
            acc0 = __builtin_amdgcn_mfma_f32_16x16x32_f16(af, __builtin_bit_cast(f16x8, wf[s][0]), acc0, 0, 0, 0); \
            acc1 = __builtin_amdgcn_mfma_f32_16x16x32_f16(af, __builtin_bit_cast(f16x8, wf[s][1]), acc1, 0, 0, 0); \
        } \
        char* red = smem + LDS_XS + (st) * 16384; \
        _Pragma("unroll") for (int r = 0; r < 4; ++r) { \
            f32x2 v = {acc0[r], acc1[r]}; \
            *(f32x2*)(red + wave * 2048 + (lg * 4 + r) * 128 + (lane & 7) * 16 + ((lane >> 3) & 1) * 8) = v; \
        } \
        asm volatile("s_waitcnt lgkmcnt(0)\n\ts_barrier" ::: "memory"); \
        { \
            const int o = wave * 256 + li * 16; \
            f32x4 s = *(const f32x4*)(red + o); \
            _Pragma("unroll") for (int w = 1; w < 8; ++w) s += *(const f32x4*)(red + w * 2048 + o); \
            s[0] = fmaxf(s[0], 0.f); s[1] = fmaxf(s[1], 0.f); s[2] = fmaxf(s[2], 0.f); s[3] = fmaxf(s[3], 0.f); \
            __builtin_amdgcn_raw_buffer_store_b128(__builtin_bit_cast(u32x4, s), orr, ooff, (st) * 16 * NB * 4, AUX_SYS); \
        } } while (0)
    const unsigned ooff = lane < 16 ? (unsigned)((P.rank * ML + m0 + wave * 2 + (lane >> 3)) * NB + nt * 32 + (lane & 7) * 4) * 4u : 0x80000000u;

    STAGE(0);
    STAMP(2);
    SB();
    STX(xa);
    SB();
    LDX(xa, 3);
    SB();
    STAGE(1);
    STAMP(3);
    SB();
    STX(xb);
    STAGE(2);
    STAMP(4);
    SB();
    STX(xa);
    STAGE(3);
    STAMP(5);

    asm volatile("s_waitcnt vmcnt(0)\n\ts_barrier" ::: "memory");
    STAMP(6);
    if (wave == 0) {
        call = __builtin_amdgcn_readfirstlane(call) + 1u;
        if (lane == 0) {
            if (dst != (unsigned)P.rank)
                __hip_atomic_store(P.ws[dst] + WS_FLAG / 4 + P.rank * WG_PER_DST + idx, call,
                                   __ATOMIC_RELAXED, __HIP_MEMORY_SCOPE_SYSTEM);
            __hip_atomic_store(myws + WS_SEQ / 4 + b, call, __ATOMIC_RELAXED, __HIP_MEMORY_SCOPE_AGENT);
        }
        if (dst == (unsigned)P.rank && idx == 0) {
            const __amdgpu_buffer_rsrc_t fr = __builtin_amdgcn_make_buffer_rsrc((void*)(myws + WS_FLAG / 4), 0, NRANK * 128, 0x00020000);
            const bool self = (lane >> 3) == P.rank;
            const unsigned long long tw = __builtin_amdgcn_s_memrealtime();
            bool timed_out = false;
            for (;;) {
                asm volatile("" ::: "memory");
                u32x4 f = __builtin_amdgcn_raw_buffer_load_b128(fr, lane * 16, 0, AUX_SYS);
                bool ok = self || (f[0] >= call && f[1] >= call && f[2] >= call && f[3] >= call);
                if (__all(ok)) break;
                if (__builtin_amdgcn_s_memrealtime() - tw > DEADLINE_TICKS) { timed_out = true; break; }
                __builtin_amdgcn_s_sleep(1);
            }
            if (timed_out && lane == 0) {
                __hip_atomic_store(myws + WS_TMO / 4, call, __ATOMIC_RELAXED, __HIP_MEMORY_SCOPE_AGENT);
                P.out[P.rank][(size_t)(P.rank * ML + m0) * NB + nt * 32] = __builtin_nanf("");
            }
            __builtin_amdgcn_fence(__ATOMIC_ACQUIRE, "");
        }
    }
}

extern "C" void kernel_launch(int rank, int nranks, ncclComm_t comm,
                              hipStream_t stream,
                              void* const* d_in, const int* in_sizes, int n_in,
                              void* d_out, int out_size,
                              const int* in_dtypes, int out_dtype,
                              void* const* d_out_all,
                              void* const* d_ws_all, size_t ws_bytes) {
    if (nranks != NRANK || (unsigned)rank >= (unsigned)NRANK || ws_bytes < WS_BYTES_USED) {
        fprintf(stderr, "unexpected configuration\n");
        return;
    }
    Params P;
    P.x = (const float*)d_in[0];
    P.w = (const float*)d_in[1];
    for (int p = 0; p < NRANK; ++p) {
        P.out[p] = (float*)d_out_all[p];
        P.ws[p] = (unsigned*)d_ws_all[p];
    }
    P.rank = rank;
    HIP_CHECK(hipFuncSetAttribute((const void*)gemm_a2a_kernel, hipFuncAttributeMaxDynamicSharedMemorySize, LDS_BYTES));
    hipLaunchKernelGGL(gemm_a2a_kernel, dim3(NWG), dim3(512), LDS_BYTES, stream, P);
    HIP_CHECK(hipGetLastError());
}
```

```cpp
#include <hip/hip_runtime.h>
#include <rccl/rccl.h>
#include <cstdio>
#include <cstdint>

#define HIP_CHECK(expr) do { hipError_t _e = (expr); if (_e != hipSuccess) { \
    fprintf(stderr, "HIP error %s at %s:%d\n", hipGetErrorString(_e), __FILE__, __LINE__); \
    return; } } while (0)

typedef _Float16 f16x8 __attribute__((ext_vector_type(8)));
typedef _Float16 f16x2 __attribute__((ext_vector_type(2)));
typedef float f32x4 __attribute__((ext_vector_type(4)));
typedef float f32x2 __attribute__((ext_vector_type(2)));
typedef unsigned u32x4 __attribute__((ext_vector_type(4)));
typedef unsigned u32x2 __attribute__((ext_vector_type(2)));

constexpr int NRANK = 8;
constexpr int ML = 256;
constexpr int KD = 2048;
constexpr int ND = 2048;
constexpr int NB = 256;
constexpr int NWG = 256;
constexpr int WG_PER_DST = NWG / NRANK;
constexpr int NTHREADS = 640;

constexpr unsigned WS_SEQ = 0;
constexpr unsigned WS_FLAG = 4096;
constexpr unsigned WS_TMO = 8192;
constexpr unsigned WS_X16 = 1u << 20;
constexpr unsigned WS_BYTES_USED = 3u << 20;
constexpr unsigned long long DEADLINE_TICKS = 50000000ull;
constexpr unsigned long long SELF_SERVE_TICKS = 3000ull;

struct Params {
    const float* x;
    const float* w;
    float* out[NRANK];
    unsigned* ws[NRANK];
    int rank;
};

constexpr int AUX_SYS = 17;
constexpr int AUX_DEV = 16;

__device__ __forceinline__ unsigned cvt2(float a, float b) {
    f32x2 v = {a, b};
    return __builtin_bit_cast(unsigned, __builtin_convertvector(v, f16x2));
}
__device__ __forceinline__ unsigned umin3(unsigned a, unsigned b, unsigned c) {
    return min(min(a, b), c);
}
#define SB() __builtin_amdgcn_sched_barrier(0)
#define BARRIER() do { SB(); asm volatile("s_barrier" ::: "memory"); SB(); } while (0)
#define STAMP(i)
#define XORD 9
#define ZPOS 8
#define HELP 1

constexpr int LDS_RED = 4 * 8 * 2048;
constexpr int LDS_PAR = LDS_RED;
constexpr int LDS_BYTES = LDS_RED + 128;

__global__ void __launch_bounds__(NTHREADS) gemm_a2a_kernel(Params P) {
    extern __shared__ __attribute__((aligned(16))) char smem[];
    const int tid = threadIdx.x;
    const int lane = tid & 63;
    const int wave = __builtin_amdgcn_readfirstlane(tid >> 6) - 2;
    const int b = blockIdx.x;
    const int grp = b & 7, idx = b >> 3;
    const int mt = idx & 3, nv = idx >> 2;
#define PHYS(v) (((0x26543710u >> (4 * (v))) + wsh) & 7u)
    const int m0 = mt * 64;
    STAMP(0);

    unsigned* myws = P.ws[0];
    const float* xp = P.x;
    const float* wp = P.w;
    float* op = P.out[grp];
    int rank = P.rank;
    asm volatile("" : "+s"(myws), "+s"(xp), "+s"(wp), "+s"(op), "+s"(rank));
    const unsigned dst = (unsigned)(grp + rank) & 7u;
    const unsigned wsh = (0u - (unsigned)((uintptr_t)wp >> 7)) & 7u;
    const int nt = PHYS(nv);
    const int n0 = dst * NB + nt * 32;
    const __amdgpu_buffer_rsrc_t hr = __builtin_amdgcn_make_buffer_rsrc((void*)(myws + WS_X16 / 4), 0, 2u << 20, 0x00020000);
    const __amdgpu_buffer_rsrc_t xr = __builtin_amdgcn_make_buffer_rsrc((void*)xp, 0, ML * KD * 4, 0x00020000);
    const __amdgpu_buffer_rsrc_t wr = __builtin_amdgcn_make_buffer_rsrc((void*)wp, 0, KD * ND * 4, 0x00020000);

    const int prb = ((b >> 4) & 3) * 4 + (b >> 6);
    const int pks = (b & 15) * 4 + (wave + 2) * 2;
    f32x4 px[4];
    unsigned callv = 0, pf = 0;
    if (wave < 0) {
        const unsigned poff = (unsigned)((prb * 16 + (lane >> 3)) * KD + pks * 32 + (lane & 7) * 4) * 4u;
#pragma unroll
        for (int i = 0; i < 4; ++i)
            px[i] = __builtin_bit_cast(f32x4, __builtin_amdgcn_raw_buffer_load_b128(xr, poff + (i >> 1) * 128, (i & 1) * 8 * KD * 4, 0));
        const __amdgpu_buffer_rsrc_t sr = __builtin_amdgcn_make_buffer_rsrc((void*)myws, 0, 4096, 0x00020000);
        callv = __builtin_amdgcn_raw_buffer_load_b32(sr, 0, WS_SEQ + b * 4, AUX_DEV);
    }
    else if (nv >= 4 && wave < 6) {
        const int j = nv - 4, o = j * 4 + mt;
        const unsigned vq = (unsigned)(((lane >> 3) * 256 + (lane & 7)) * ND + dst * NB) * 4u;
        const int t = 3 - (wave < 4 ? (wave >> 1) : wave - 2);
        const int row = wave < 4 ? (o >> 1) * 32 + ((o & 1) * 2 + (wave & 1)) * 8 : ((2 * j + t + 1) & 7) * 32 + mt * 8;
        pf = __builtin_amdgcn_raw_buffer_load_b32(wr, vq, (unsigned)row * ND * 4u + PHYS(t) * 128, 0);
    }
    BARRIER();

    if (__builtin_expect(wave >= 0, 1)) {
        const int kw = wave * 256;
        const int li = lane & 15, lg = lane >> 4;
        const unsigned woff = (unsigned)((kw + 4 * (lane >> 3)) * ND + n0 + 4 * (lane & 7)) * 4u;
        const int rot = (2 * mt + nv) & 7;
        STAMP(25);

        f32x4 wv[8][4];
        u32x4 wf[8][2];
        u32x4 xf[4][8];
#define LDW1(i) wv[(i) >> 2][(i) & 3] = __builtin_bit_cast(f32x4, __builtin_amdgcn_raw_buffer_load_b128(wr, woff, (((((i) >> 2) + rot) & 7) * 32 + ((i) & 3)) * ND * 4, 0))
#define LDWG(s) do { LDW1(4 * (s)); LDW1(4 * (s) + 1); LDW1(4 * (s) + 2); LDW1(4 * (s) + 3); } while (0)
#define CVW(s) _Pragma("unroll") for (int t = 0; t < 2; ++t) _Pragma("unroll") for (int d = 0; d < 2; ++d) { \
        unsigned p0 = cvt2(wv[s][2 * d][t], wv[s][2 * d + 1][t]); \
        unsigned p2 = cvt2(wv[s][2 * d][2 + t], wv[s][2 * d + 1][2 + t]); \
        wf[s][t][d] = __builtin_amdgcn_update_dpp(p0, p2, 0x128, 0xf, 0xc, false); \
        wf[s][t][2 + d] = __builtin_amdgcn_update_dpp(p2, p0, 0x128, 0xf, 0x3, false); }
#define LDXH(st, h) _Pragma("unroll") for (int s = 4 * (h); s < 4 * (h) + 4; ++s) \
        xf[st][s] = __builtin_amdgcn_raw_buffer_load_b128(hr, hoff, hbase + (st) * 65536 + ((s + rot) & 7) * 1024, AUX_DEV)
#define LDX(st) do { LDXH(st, 0); LDXH(st, 1); } while (0)
#define WAITW(s) do { SB(); asm volatile("" :: "v"(wv[s][3])); SB(); } while (0)

        LDWG(0); LDWG(1); LDWG(2);
        const unsigned hoff = (unsigned)(lane * 16 + wave * 8192);
        WAITW(0); BARRIER(); STAMP(17); LDWG(3); SB(); CVW(0);
        asm volatile("" :: "v"(pf));
        WAITW(1); BARRIER(); STAMP(18); LDWG(4); SB(); CVW(1);
        WAITW(2); BARRIER(); STAMP(19); LDWG(5); SB(); CVW(2);
        WAITW(3); BARRIER(); STAMP(20); LDWG(6); SB(); CVW(3);
        WAITW(4); BARRIER(); STAMP(21); LDWG(7); SB(); CVW(4);
        const unsigned par = __builtin_amdgcn_readfirstlane(*(const unsigned*)(smem + LDS_PAR));
        const unsigned hbase = (par << 20) + (unsigned)mt * 262144u;
        WAITW(5); BARRIER(); STAMP(22); LDXH(0, 0); SB(); CVW(5);
        WAITW(6); BARRIER(); STAMP(23); LDXH(0, 1); SB(); CVW(6);
        WAITW(7); CVW(7); BARRIER(); STAMP(24); LDXH(1, 0);
        SB();
        STAMP(2);

#define CNT_RETRY
#define XMIN(st, m) do { m = 0xffffffffu; \
        _Pragma("unroll") for (int s = 0; s < 8; ++s) { \
            m = umin3(m, xf[st][s][0], xf[st][s][1]); m = umin3(m, xf[st][s][2], xf[st][s][3]); } } while (0)
#define STAGE(st, PREBAR, POST) do { \
        f32x4 acc0 = {0.f, 0.f, 0.f, 0.f}, acc1 = {0.f, 0.f, 0.f, 0.f}; \
        unsigned m; \
        _Pragma("unroll") for (int s = 0; s < 8; ++s) { \
            f16x8 af = __builtin_bit_cast(f16x8, xf[st][s]); \
            acc0 = __builtin_amdgcn_mfma_f32_16x16x32_f16(af, __builtin_bit_cast(f16x8, wf[s][0]), acc0, 0, 0, 0); \
            acc1 = __builtin_amdgcn_mfma_f32_16x16x32_f16(af, __builtin_bit_cast(f16x8, wf[s][1]), acc1, 0, 0, 0); \
        } \
        XMIN(st, m); \
        if (st == 0) STAMP(12); \
        if (__builtin_expect(__any(m == 0u), 0)) { \
              \
            const unsigned long long tw = __builtin_amdgcn_s_memrealtime(); \
            const unsigned fo = (unsigned)((m0 + (st) * 16 + li) * KD + kw + lg * 8) * 4u; \
            acc0 = f32x4{0.f, 0.f, 0.f, 0.f}; acc1 = f32x4{0.f, 0.f, 0.f, 0.f}; \
            CNT_RETRY; \
            _Pragma("unroll") for (int s = 0; s < 8; ++s) { \
                u32x4 f; \
                for (;;) { \
                    asm volatile("" ::: "memory"); \
                    f = __builtin_amdgcn_raw_buffer_load_b128(hr, hoff, hbase + (st) * 65536 + ((s + rot) & 7) * 1024, AUX_DEV); \
                    if (!__any(min(min(f[0], f[1]), min(f[2], f[3])) == 0u)) break; \
                    if (__builtin_amdgcn_s_memrealtime() - tw > SELF_SERVE_TICKS) { \
                        f32x4 qa = __builtin_bit_cast(f32x4, __builtin_amdgcn_raw_buffer_load_b128(xr, fo, ((s + rot) & 7) * 128, 0)); \
                        f32x4 qb = __builtin_bit_cast(f32x4, __builtin_amdgcn_raw_buffer_load_b128(xr, fo + 16, ((s + rot) & 7) * 128, 0)); \
                        f = u32x4{cvt2(qa[0], qa[1]), cvt2(qa[2], qa[3]), cvt2(qb[0], qb[1]), cvt2(qb[2], qb[3])}; \
                        break; \
                    } \
                    __builtin_amdgcn_s_sleep(1); \
                } \
                f16x8 af = __builtin_bit_cast(f16x8, f); \
                acc0 = __builtin_amdgcn_mfma_f32_16x16x32_f16(af, __builtin_bit_cast(f16x8, wf[s][0]), acc0, 0, 0, 0); \
                acc1 = __builtin_amdgcn_mfma_f32_16x16x32_f16(af, __builtin_bit_cast(f16x8, wf[s][1]), acc1, 0, 0, 0); \
                SB(); \
            } \
        } \
        char* red = smem + (st) * 16384; \
        _Pragma("unroll") for (int r = 0; r < 4; ++r) { \
            f32x2 v = {acc0[r], acc1[r]}; \
            *(f32x2*)(red + wave * 2048 + (lg * 4 + r) * 128 + (lane & 7) * 16 + ((lane >> 3) & 1) * 8) = v; \
        } \
        SB(); PREBAR; SB(); \
        asm volatile("s_waitcnt lgkmcnt(0)\n\ts_barrier" ::: "memory"); \
        SB(); if (st == 0) STAMP(11); POST; SB(); \
        } while (0)

        STAGE(0, LDXH(1, 1), LDXH(2, 0));
        STAGE(1, LDXH(2, 1), LDXH(3, 0));
        STAGE(2, LDXH(3, 1), );
        STAGE(3, , );
        return;
    }

    const int pw = wave + 2;
    __builtin_amdgcn_s_setprio(3);
    const unsigned call = __builtin_amdgcn_readfirstlane(callv) + 1u;
    {
        char* tb = smem + 3 * 16384 + pw * 2048;
        const int c = lane & 7;
#pragma unroll
        for (int i = 0; i < 4; ++i) {
            u32x2 h = {cvt2(px[i][0], px[i][1]), cvt2(px[i][2], px[i][3])};
            *(u32x2*)(tb + (i >> 1) * 1024 + ((i & 1) * 8 + (lane >> 3) + 16 * (c >> 1)) * 16 + (c & 1) * 8) = h;
        }
        STAMP(10);
        const unsigned par = call & 1u;
        if (pw == 0 && lane == 0) *(unsigned*)(smem + LDS_PAR) = par;
        asm volatile("s_waitcnt lgkmcnt(0)" ::: "memory");
        const unsigned so = (unsigned)(prb * 64 + pks) * 1024u;
#pragma unroll
        for (int j = 0; j < 2; ++j) {
            u32x4 h = *(const u32x4*)(tb + j * 1024 + lane * 16);
#pragma unroll
            for (int i = 0; i < 4; ++i) h[i] = h[i] == 0u ? 0x8000u : h[i];
            __builtin_amdgcn_raw_buffer_store_b128(h, hr, lane * 16 + j * 1024, so + (par << 20), AUX_DEV);
        }
        STAMP(1);
        const u32x4 z = {0u, 0u, 0u, 0u};
#pragma unroll
        for (int g = 0; g < 8; ++g) {
            BARRIER();
            if (g == ZPOS) {
                __builtin_amdgcn_raw_buffer_store_b128(z, hr, lane * 16, so + ((par ^ 1u) << 20), AUX_DEV);
                __builtin_amdgcn_raw_buffer_store_b128(z, hr, lane * 16 + 1024, so + ((par ^ 1u) << 20), AUX_DEV);
            }
        }
    }
    const __amdgpu_buffer_rsrc_t orr = __builtin_amdgcn_make_buffer_rsrc((void*)op, 0, ND * NB * 4, 0x00020000);
    const unsigned ooff = (unsigned)((rank * ML + m0 + pw * 8 + (lane >> 3)) * NB + nt * 32 + (lane & 7) * 4) * 4u;
#pragma unroll
    for (int st = 0; st < 4; ++st) {
        BARRIER();
        const char* red = smem + st * 16384 + pw * 1024 + lane * 16;
        f32x4 v[8];
#pragma unroll
        for (int w = 0; w < 8; ++w) v[w] = *(const f32x4*)(red + w * 2048);
        f32x4 s = ((v[0] + v[1]) + (v[2] + v[3])) + ((v[4] + v[5]) + (v[6] + v[7]));
        s[0] = fmaxf(s[0], 0.f); s[1] = fmaxf(s[1], 0.f); s[2] = fmaxf(s[2], 0.f); s[3] = fmaxf(s[3], 0.f);
        __builtin_amdgcn_raw_buffer_store_b128(__builtin_bit_cast(u32x4, s), orr, ooff, st * 16 * NB * 4, AUX_SYS);
        STAMP(3 + st);
    }
    if (ZPOS == 8) {
        const u32x4 z = {0u, 0u, 0u, 0u};
        const unsigned so = (unsigned)(prb * 64 + pks) * 1024u + (((call & 1u) ^ 1u) << 20);
        __builtin_amdgcn_raw_buffer_store_b128(z, hr, lane * 16, so, AUX_DEV);
        __builtin_amdgcn_raw_buffer_store_b128(z, hr, lane * 16 + 1024, so, AUX_DEV);
        asm volatile("s_waitcnt vmcnt(2)\n\ts_barrier" ::: "memory");
    } else
    asm volatile("s_waitcnt vmcnt(0)\n\ts_barrier" ::: "memory");
    STAMP(7);
    if (pw == 0) {
        if (lane == 0) {
            if (grp != 0)
                __hip_atomic_store(P.ws[grp] + WS_FLAG / 4 + rank * WG_PER_DST + idx, call,
                                   __ATOMIC_RELAXED, __HIP_MEMORY_SCOPE_SYSTEM);
            __hip_atomic_store(P.ws[0] + WS_SEQ / 4 + b, call, __ATOMIC_RELAXED, __HIP_MEMORY_SCOPE_AGENT);
        }
        if (b == 0) {
            const __amdgpu_buffer_rsrc_t fr = __builtin_amdgcn_make_buffer_rsrc((void*)(myws + WS_FLAG / 4), 0, NRANK * 128, 0x00020000);
            const bool self = (lane >> 3) == rank;
            const unsigned long long tw = __builtin_amdgcn_s_memrealtime();
            bool timed_out = false;
            for (;;) {
                asm volatile("" ::: "memory");
                u32x4 f = __builtin_amdgcn_raw_buffer_load_b128(fr, lane * 16, 0, AUX_SYS);
                bool ok = self || (f[0] >= call && f[1] >= call && f[2] >= call && f[3] >= call);
                if (__all(ok)) break;
                if (__builtin_amdgcn_s_memrealtime() - tw > DEADLINE_TICKS) { timed_out = true; break; }
                __builtin_amdgcn_s_sleep(1);
            }
            if (timed_out && lane == 0) {
                __hip_atomic_store(P.ws[0] + WS_TMO / 4, call, __ATOMIC_RELAXED, __HIP_MEMORY_SCOPE_AGENT);
                P.out[0][(size_t)(rank * ML + m0) * NB + nt * 32] = __builtin_nanf("");
            }
            __builtin_amdgcn_fence(__ATOMIC_ACQUIRE, "");
        }
    }
}

extern "C" void kernel_launch(int rank, int nranks, ncclComm_t comm,
                              hipStream_t stream,
                              void* const* d_in, const int* in_sizes, int n_in,
                              void* d_out, int out_size,
                              const int* in_dtypes, int out_dtype,
                              void* const* d_out_all,
                              void* const* d_ws_all, size_t ws_bytes) {
    if (nranks != NRANK || (unsigned)rank >= (unsigned)NRANK || ws_bytes < WS_BYTES_USED) {
        fprintf(stderr, "unexpected configuration\n");
        return;
    }
    Params P;
    P.x = (const float*)d_in[0];
    P.w = (const float*)d_in[1];
    for (int p = 0; p < NRANK; ++p) {
        P.out[p] = (float*)d_out_all[(p + rank) & 7];
        P.ws[p] = (unsigned*)d_ws_all[(p + rank) & 7];
    }
    P.rank = rank;
    HIP_CHECK(hipFuncSetAttribute((const void*)gemm_a2a_kernel, hipFuncAttributeMaxDynamicSharedMemorySize, LDS_BYTES));
    hipLaunchKernelGGL(gemm_a2a_kernel, dim3(NWG), dim3(NTHREADS), LDS_BYTES, stream, P);
    HIP_CHECK(hipGetLastError());
}
```

```cpp
#include <hip/hip_runtime.h>
#include <rccl/rccl.h>
#include <cstdio>
#include <cstdint>

#define HIP_CHECK(expr) do { hipError_t _e = (expr); if (_e != hipSuccess) { \
    fprintf(stderr, "HIP error %s at %s:%d\n", hipGetErrorString(_e), __FILE__, __LINE__); \
    return; } } while (0)

typedef _Float16 f16x8 __attribute__((ext_vector_type(8)));
typedef _Float16 f16x2 __attribute__((ext_vector_type(2)));
typedef float f32x4 __attribute__((ext_vector_type(4)));
typedef float f32x2 __attribute__((ext_vector_type(2)));
typedef unsigned u32x4 __attribute__((ext_vector_type(4)));
typedef unsigned u32x2 __attribute__((ext_vector_type(2)));
typedef unsigned u32x8 __attribute__((ext_vector_type(8)));

constexpr int NRANK = 8;
constexpr int ML = 256;
constexpr int KD = 2048;
constexpr int ND = 2048;
constexpr int NB = 256;
constexpr int NWG = 256;
constexpr int WG_PER_DST = NWG / NRANK;
constexpr int NTHREADS = 640;

constexpr unsigned WS_SEQ = 0;
constexpr unsigned WS_FLAG = 4096;
constexpr unsigned WS_TMO = 8192;
constexpr unsigned WS_X16 = 1u << 20;
constexpr unsigned WS_BYTES_USED = 3u << 20;
constexpr unsigned long long DEADLINE_TICKS = 50000000ull;
constexpr unsigned long long SELF_SERVE_TICKS = 3000ull;

#define LOCALG 4
struct __attribute__((aligned(128))) GroupParams {
    const char* wb[8];
    char* out;
    unsigned* flag;
    const float* x;
    unsigned* ws;
    unsigned ptab;
    unsigned rank;
    unsigned long long* dbg;
    unsigned pad[4];
};
struct Params { GroupParams g[NRANK]; };

constexpr int AUX_SYS = 17;
constexpr int AUX_DEV = 16;

__device__ __forceinline__ unsigned cvt2(float a, float b) {
    f32x2 v = {a, b};
    return __builtin_bit_cast(unsigned, __builtin_convertvector(v, f16x2));
}
__device__ __forceinline__ unsigned umin3(unsigned a, unsigned b, unsigned c) {
    return min(min(a, b), c);
}
#define SB() __builtin_amdgcn_sched_barrier(0)
#define BARRIER() do { SB(); asm volatile("s_barrier" ::: "memory"); SB(); } while (0)
#define STAMP(i)
#define XORD 9
#define PM 0xffffffffu
#define PML 0u
#define POLL 0
#define PSL 2
#define FAUXS "sc0 sc1"
#define FAUX AUX_SYS
#define CODEWARM 0
#define CONV 0
#define HELP 1

constexpr int LDS_RED = 4 * 8 * 2048;
constexpr int LDS_PAR = LDS_RED;
constexpr int LDS_BYTES = LDS_RED + 128;

__global__ void __launch_bounds__(NTHREADS) gemm_a2a_kernel(Params P) {
    extern __shared__ __attribute__((aligned(16))) char smem[];
    const int tid = threadIdx.x;
    const int lane = tid & 63;
    const int wave = __builtin_amdgcn_readfirstlane(tid >> 6) - 2;
    const int b = blockIdx.x;
    const int grp = b & 7, idx = b >> 3;
    const int mt = idx & 3, nv = idx >> 2;
    const int m0 = mt * 64;
    STAMP(0);
    const char* ka = (const char*)__builtin_amdgcn_kernarg_segment_ptr() + grp * 128;
    unsigned long long pc;
    asm volatile("s_load_dword s100, %1, 0x0\n\ts_load_dword s101, %1, 0x40\n\ts_getpc_b64 %0" : "=s"(pc) : "s"(ka) : "s100", "s101", "memory");
    SB();
#define PTR(lo, hi) ((void*)(((unsigned long long)(hi) << 32) | (lo)))
#define RSRC(lo, hi, bytes) __builtin_amdgcn_make_buffer_rsrc(PTR(lo, hi), 0, bytes, 0x00020000)
    constexpr unsigned W_SPAN = (KD - 1) * ND * 4 + 128;
#define CLOAD(u, rb) _Pragma("unroll") for (int i = 0; i < 4; ++i) \
        px[4 * (u) + i] = __builtin_bit_cast(f32x4, __builtin_amdgcn_raw_buffer_load_b128(xr, cvo + (i >> 1) * 128, ((i & 1) * 8 + (rb) * 16) * KD * 4, 0))
#define CONVERT(u, rb) do { \
        const unsigned so = WS_X16 + (((__builtin_amdgcn_readfirstlane(callv) + 1u) & 1u) << 20) + (unsigned)((crb + (rb)) * 64 + cks) * 1024u; \
        const int c = lane & 7; \
        char* tb = smem + (wave + 2) * 4096 + (u) * 2048; \
        _Pragma("unroll") for (int i = 0; i < 4; ++i) { \
            u32x2 h = {cvt2(px[4 * (u) + i][0], px[4 * (u) + i][1]), cvt2(px[4 * (u) + i][2], px[4 * (u) + i][3])}; \
            *(u32x2*)(tb + (i >> 1) * 1024 + ((i & 1) * 8 + (lane >> 3) + 16 * (c >> 1)) * 16 + (c & 1) * 8) = h; \
        } \
        asm volatile("s_waitcnt lgkmcnt(0)" ::: "memory"); \
        _Pragma("unroll") for (int j = 0; j < 2; ++j) { \
            u32x4 h = *(const u32x4*)(tb + j * 1024 + lane * 16); \
            _Pragma("unroll") for (int i = 0; i < 4; ++i) h[i] = h[i] == 0u ? 0x8000u : h[i]; \
            asm volatile("buffer_store_dwordx4 %0, %1, %2, %3 offen sc1\n\ts_nop 1" :: "v"(h), "v"(lane * 16 + j * 1024), "s"(sr), "s"(so) : "memory"); \
        } \
        SB(); } while (0)

    if (__builtin_expect(wave >= 0, 1)) {
        if (__builtin_expect(grp == LOCALG, 0)) {
            u32x4 xw;
            asm volatile("s_load_dwordx4 %0, %1, 0x50" : "=&s"(xw) : "s"(ka) : "memory");
            const int crb = (idx >> 3) * 4 + (CONV == 0 ? 0 : 2) + (wave >> 2), cks = ((idx & 7) * 4 + (wave & 3)) * 2;
            const unsigned cvo = (unsigned)((crb * 16 + (lane >> 3)) * KD + cks * 32 + (lane & 7) * 4) * 4u;
            asm volatile("s_waitcnt lgkmcnt(0)" : "+s"(xw) : "v"(cvo) : "memory");
            const __amdgpu_buffer_rsrc_t xr = RSRC(xw[0], xw[1], ML * KD * 4), sr = RSRC(xw[2], xw[3], WS_BYTES_USED);
            f32x4 px[8];
            const unsigned callv = __builtin_amdgcn_raw_buffer_load_b32(sr, 0, WS_SEQ + b * 4, AUX_DEV);
            CLOAD(0, 0);
            BARRIER();
            CLOAD(1, 2);
            SB();
            CONVERT(0, 0);
            STAMP(13);
            CONVERT(1, 2);
            STAMP(14);
        }
        u32x2 wown, whelp;
        u32x4 xw;
        const int ht = (0x00012233u >> (4 * wave)) & 3;
        asm volatile("s_load_dwordx2 %0, %3, %4\n\ts_load_dwordx2 %1, %3, %5\n\ts_load_dwordx4 %2, %3, 0x50"
                     : "=&s"(wown), "=&s"(whelp), "=&s"(xw) : "s"(ka), "s"(nv * 8), "s"(ht * 8) : "memory");
        const int kw = wave * 256;
        const int rot = (2 * mt + nv) & 7;
        const unsigned woff = (unsigned)((kw + 4 * (lane >> 3)) * ND + 4 * (lane & 7)) * 4u;
        const unsigned wso0 = (unsigned)rot * (32 * ND * 4);
        const unsigned hvo = (unsigned)(((lane >> 3) * 256 + (lane & 7)) * ND) * 4u;
        const unsigned hr0 = (idx & 15) * 16 + (wave & 1) * 8, hr1 = ((2 * nv + ht + 1) & 7) * 32 + mt * 8;
        unsigned hso = (wave < 4 ? hr0 : hr1) * (ND * 4u);
        asm volatile("s_waitcnt lgkmcnt(0)" : "+s"(wown), "+s"(whelp), "+s"(xw) : "v"(woff), "s"(wso0), "v"(hvo), "s"(hso), "s"(hr0), "s"(hr1) : "memory");
        const __amdgpu_buffer_rsrc_t wr = RSRC(wown[0], wown[1], W_SPAN);
        unsigned pf = 0;
        if (nv >= 4 || grp == LOCALG) {
            if (nv >= 4 && wave < 6) pf = __builtin_amdgcn_raw_buffer_load_b32(RSRC(whelp[0], whelp[1], W_SPAN), hvo, hso, 0);
            BARRIER();
        }
        const __amdgpu_buffer_rsrc_t xr = RSRC(xw[0], xw[1], ML * KD * 4), sr = RSRC(xw[2], xw[3], WS_BYTES_USED);
        const int li = lane & 15, lg = lane >> 4;
        STAMP(25);

        f32x4 wv[8][4];
        u32x4 wf[8][2];
        u32x4 xf[4][8];
#define LDW1(i) wv[(i) >> 2][(i) & 3] = __builtin_bit_cast(f32x4, __builtin_amdgcn_raw_buffer_load_b128(wr, woff, (((((i) >> 2) + rot) & 7) * 32 + ((i) & 3)) * ND * 4, 0))
#define LDWG(s) do { LDW1(4 * (s)); LDW1(4 * (s) + 1); LDW1(4 * (s) + 2); LDW1(4 * (s) + 3); } while (0)
#define CVW(s) _Pragma("unroll") for (int t = 0; t < 2; ++t) _Pragma("unroll") for (int d = 0; d < 2; ++d) { \
        unsigned p0 = cvt2(wv[s][2 * d][t], wv[s][2 * d + 1][t]); \
        unsigned p2 = cvt2(wv[s][2 * d][2 + t], wv[s][2 * d + 1][2 + t]); \
        wf[s][t][d] = __builtin_amdgcn_update_dpp(p0, p2, 0x128, 0xf, 0xc, false); \
        wf[s][t][2 + d] = __builtin_amdgcn_update_dpp(p2, p0, 0x128, 0xf, 0x3, false); }
#define LDXH(st, h) _Pragma("unroll") for (int s = 4 * (h); s < 4 * (h) + 4; ++s) \
        xf[st][s] = __builtin_amdgcn_raw_buffer_load_b128(sr, hoff, hbase + (st) * 65536 + ((s + rot) & 7) * 1024, AUX_DEV)
#define LDX(st) do { LDXH(st, 0); LDXH(st, 1); } while (0)
#define WAITW(s) do { SB(); asm volatile("" :: "v"(wv[s][3])); SB(); } while (0)

        LDWG(0); LDWG(1); LDWG(2);
        const unsigned hoff = (unsigned)(lane * 16 + wave * 8192);
        WAITW(0); BARRIER(); STAMP(17); LDWG(3); SB(); CVW(0);
        asm volatile("" :: "v"(pf));
        WAITW(1); BARRIER(); STAMP(18); LDWG(4); SB(); CVW(1);
        WAITW(2); BARRIER(); STAMP(19); LDWG(5); SB(); CVW(2);
        WAITW(3); BARRIER(); STAMP(20); LDWG(6); SB(); CVW(3);
        WAITW(4); BARRIER(); STAMP(21); LDWG(7); SB(); CVW(4);
        const unsigned par = __builtin_amdgcn_readfirstlane(*(const unsigned*)(smem + LDS_PAR));
        const unsigned hbase = WS_X16 + (par << 20) + (unsigned)mt * 262144u;
        WAITW(5); BARRIER(); STAMP(22); LDXH(0, 0); SB(); CVW(5);
        WAITW(6); BARRIER(); STAMP(23); LDXH(0, 1); SB(); CVW(6);
        WAITW(7); CVW(7); BARRIER(); STAMP(24); LDXH(1, 0);
        SB();
        STAMP(2);

#define CNT_RETRY
#define XMIN(st, m) do { m = 0xffffffffu; \
        _Pragma("unroll") for (int s = 0; s < 8; ++s) { \
            m = umin3(m, xf[st][s][0], xf[st][s][1]); m = umin3(m, xf[st][s][2], xf[st][s][3]); } } while (0)
#define STAGE(st, PREBAR, POST) do { \
        f32x4 acc0 = {0.f, 0.f, 0.f, 0.f}, acc1 = {0.f, 0.f, 0.f, 0.f}; \
        unsigned m; \
        _Pragma("unroll") for (int s = 0; s < 8; ++s) { \
            f16x8 af = __builtin_bit_cast(f16x8, xf[st][s]); \
            acc0 = __builtin_amdgcn_mfma_f32_16x16x32_f16(af, __builtin_bit_cast(f16x8, wf[s][0]), acc0, 0, 0, 0); \
            acc1 = __builtin_amdgcn_mfma_f32_16x16x32_f16(af, __builtin_bit_cast(f16x8, wf[s][1]), acc1, 0, 0, 0); \
        } \
        XMIN(st, m); \
        if (st == 0) STAMP(12); \
        if (__builtin_expect(__any(m == 0u), 0)) { \
              \
            const unsigned long long tw = __builtin_amdgcn_s_memrealtime(); \
            const unsigned fo = (unsigned)((m0 + (st) * 16 + li) * KD + kw + lg * 8) * 4u; \
            acc0 = f32x4{0.f, 0.f, 0.f, 0.f}; acc1 = f32x4{0.f, 0.f, 0.f, 0.f}; \
            CNT_RETRY; \
            _Pragma("unroll") for (int s = 0; s < 8; ++s) { \
                u32x4 f; \
                for (;;) { \
                    asm volatile("" ::: "memory"); \
                    f = __builtin_amdgcn_raw_buffer_load_b128(sr, hoff, hbase + (st) * 65536 + ((s + rot) & 7) * 1024, AUX_DEV); \
                    if (!__any(min(min(f[0], f[1]), min(f[2], f[3])) == 0u)) break; \
                    if (__builtin_amdgcn_s_memrealtime() - tw > SELF_SERVE_TICKS) { \
                        f32x4 qa = __builtin_bit_cast(f32x4, __builtin_amdgcn_raw_buffer_load_b128(xr, fo, ((s + rot) & 7) * 128, 0)); \
                        f32x4 qb = __builtin_bit_cast(f32x4, __builtin_amdgcn_raw_buffer_load_b128(xr, fo + 16, ((s + rot) & 7) * 128, 0)); \
                        f = u32x4{cvt2(qa[0], qa[1]), cvt2(qa[2], qa[3]), cvt2(qb[0], qb[1]), cvt2(qb[2], qb[3])}; \
                        break; \
                    } \
                    __builtin_amdgcn_s_sleep(1); \
                } \
                f16x8 af = __builtin_bit_cast(f16x8, f); \
                acc0 = __builtin_amdgcn_mfma_f32_16x16x32_f16(af, __builtin_bit_cast(f16x8, wf[s][0]), acc0, 0, 0, 0); \
                acc1 = __builtin_amdgcn_mfma_f32_16x16x32_f16(af, __builtin_bit_cast(f16x8, wf[s][1]), acc1, 0, 0, 0); \
                SB(); \
            } \
        } \
        char* red = smem + (st) * 16384; \
        _Pragma("unroll") for (int r = 0; r < 4; ++r) { \
            f32x2 v = {acc0[r], acc1[r]}; \
            *(f32x2*)(red + wave * 2048 + (lg * 4 + r) * 128 + (lane & 7) * 16 + ((lane >> 3) & 1) * 8) = v; \
        } \
        SB(); PREBAR; SB(); \
        asm volatile("s_waitcnt lgkmcnt(0)\n\ts_barrier" ::: "memory"); \
        SB(); if (st == 0) STAMP(11); POST; SB(); \
        } while (0)

        STAGE(0, LDXH(1, 1), LDXH(2, 0));
        STAGE(1, LDXH(2, 1), LDXH(3, 0));
        STAGE(2, LDXH(3, 1), );
        STAGE(3, , );
        return;
    }

    const int pw = wave + 2;
    __builtin_amdgcn_s_setprio(3);
    u32x2 misc;
    u32x8 ptrs;
    asm volatile("s_load_dwordx8 %0, %2, 0x40\n\ts_load_dwordx2 %1, %2, 0x60" : "=&s"(ptrs), "=&s"(misc) : "s"(ka) : "memory");
    unsigned cw = 0;
    const int crb = ((b >> 4) & 3) * 4 + ((b >> 6) & 1), cks = (b & 15) * 4 + pw * 2;
    const unsigned cvo = (unsigned)((crb * 16 + (lane >> 3)) * KD + cks * 32 + (lane & 7) * 4) * 4u;
    asm volatile("s_waitcnt lgkmcnt(0)" : "+s"(ptrs), "+s"(misc) : "v"(cvo) : "memory");
    const __amdgpu_buffer_rsrc_t xr = RSRC(ptrs[4], ptrs[5], ML * KD * 4), sr = RSRC(ptrs[6], ptrs[7], WS_BYTES_USED);
    const unsigned callv = __builtin_amdgcn_raw_buffer_load_b32(sr, 0, WS_SEQ + b * 4, AUX_DEV);
    if (grp == LOCALG) BARRIER();
    if (nv >= 4 || grp == LOCALG) BARRIER();
    STAMP(10);
    const unsigned call = __builtin_amdgcn_readfirstlane(callv) + 1u;
    asm volatile("" :: "v"(cw));
    if (pw == 0 && lane == 0) *(unsigned*)(smem + LDS_PAR) = call & 1u;
    asm volatile("s_waitcnt lgkmcnt(0)" ::: "memory");
    STAMP(1);
    const unsigned rank = misc[1] & 7u;
    const unsigned nt = (misc[0] >> (4 * nv)) & 7u;
    const __amdgpu_buffer_rsrc_t orr = __builtin_amdgcn_make_buffer_rsrc(PTR(ptrs[0], ptrs[1]), 0, ML * NB * 4, 0x00020000);
    const __amdgpu_buffer_rsrc_t fr = __builtin_amdgcn_make_buffer_rsrc(PTR(ptrs[2], ptrs[3]), 0, WG_PER_DST * 4, 0x00020000);
    const unsigned ooff = (unsigned)((m0 + pw * 8 + (lane >> 3)) * NB + nt * 32 + (lane & 7) * 4) * 4u;
    const unsigned pm = grp == LOCALG ? PML : (PM >> (4 * nv + 2 * (mt & 1))) & 3u;
#pragma unroll
    for (int g = 0; g < 8; ++g) BARRIER();
    f32x4 sv[4];
#pragma unroll
    for (int st = 0; st < 4; ++st) {
        BARRIER();
        const char* red = smem + st * 16384 + pw * 1024 + lane * 16;
        f32x4 v[8];
#pragma unroll
        for (int w = 0; w < 8; ++w) v[w] = *(const f32x4*)(red + w * 2048);
        f32x4 s = ((v[0] + v[1]) + (v[2] + v[3])) + ((v[4] + v[5]) + (v[6] + v[7]));
        s[0] = fmaxf(s[0], 0.f); s[1] = fmaxf(s[1], 0.f); s[2] = fmaxf(s[2], 0.f); s[3] = fmaxf(s[3], 0.f);
        sv[st] = s;
#define PUSH(j) __builtin_amdgcn_raw_buffer_store_b128(__builtin_bit_cast(u32x4, sv[j]), orr, ooff, (j) * 16 * NB * 4, AUX_SYS)
        if (st == 0) { if (pm & 1u) PUSH(0); }
        if (st == 1) { if (pm == 2u) PUSH(0); if (pm & 2u) PUSH(1); }
        if (st == 2) { if (pm == 0u) PUSH(0); if (!(pm & 2u)) PUSH(1); PUSH(2); }
        if (st == 3) PUSH(3);
        STAMP(3 + st);
    }
    {
        const u32x4 z = {0u, 0u, 0u, 0u};
        const unsigned so = WS_X16 + (((call & 1u) ^ 1u) << 20) + (unsigned)(b * 2 + pw) * 2048u;
        __builtin_amdgcn_raw_buffer_store_b128(z, sr, lane * 16, so, AUX_DEV);
        __builtin_amdgcn_raw_buffer_store_b128(z, sr, lane * 16 + 1024, so, AUX_DEV);
        asm volatile("s_waitcnt vmcnt(2)\n\ts_barrier" ::: "memory");
    }
    STAMP(7);
    if (pw == 0) {
        if (lane == 0) {
            if (grp != LOCALG) __builtin_amdgcn_raw_buffer_store_b32(call, fr, 0, idx * 4, AUX_SYS);
            __builtin_amdgcn_raw_buffer_store_b32(call, sr, 0, WS_SEQ + b * 4, AUX_DEV);
        }
        if (b == LOCALG) {
            const bool self = (unsigned)(lane >> 3) == rank;
            const unsigned long long tw = __builtin_amdgcn_s_memrealtime();
            bool timed_out = false;
            for (;;) {
                asm volatile("" ::: "memory");
                u32x4 f = __builtin_amdgcn_raw_buffer_load_b128(sr, lane * 16, WS_FLAG, FAUX);
                bool ok = self || (f[0] >= call && f[1] >= call && f[2] >= call && f[3] >= call);
                if (__all(ok)) break;
                if (__builtin_amdgcn_s_memrealtime() - tw > DEADLINE_TICKS) { timed_out = true; break; }
                __builtin_amdgcn_s_sleep(1);
            }
            if (timed_out && lane == 0) {
                __builtin_amdgcn_raw_buffer_store_b32(call, sr, 0, WS_TMO, AUX_DEV);
                __builtin_amdgcn_raw_buffer_store_b32(0x7fc00000u, orr, 0, nt * 128, AUX_SYS);
            }
        }
    }
}

extern "C" void kernel_launch(int rank, int nranks, ncclComm_t comm,
                              hipStream_t stream,
                              void* const* d_in, const int* in_sizes, int n_in,
                              void* d_out, int out_size,
                              const int* in_dtypes, int out_dtype,
                              void* const* d_out_all,
                              void* const* d_ws_all, size_t ws_bytes) {
    if (nranks != NRANK || (unsigned)rank >= (unsigned)NRANK || ws_bytes < WS_BYTES_USED) {
        fprintf(stderr, "unexpected configuration\n");
        return;
    }
    Params P = {};
    const char* w = (const char*)d_in[1];
    const unsigned wsh = (0u - (unsigned)((uintptr_t)w >> 7)) & 7u;
    for (int g = 0; g < NRANK; ++g) {
        const int dst = (rank + g - LOCALG) & 7;
        GroupParams& G = P.g[g];
        for (int v = 0; v < 8; ++v) {
            const unsigned phys = ((0x26543710u >> (4 * v)) + wsh) & 7u;
            G.wb[v] = w + dst * (NB * 4) + phys * 128;
            G.ptab |= phys << (4 * v);
        }
        G.out = (char*)d_out_all[dst] + (size_t)rank * ML * NB * 4;
        G.flag = (unsigned*)d_ws_all[dst] + WS_FLAG / 4 + rank * WG_PER_DST;
        G.x = (const float*)d_in[0];
        G.ws = (unsigned*)d_ws_all[rank];
        G.rank = (unsigned)rank;
        G.dbg = (unsigned long long*)((char*)d_ws_all[rank] + 65536);
    }
    HIP_CHECK(hipFuncSetAttribute((const void*)gemm_a2a_kernel, hipFuncAttributeMaxDynamicSharedMemorySize, LDS_BYTES));
    hipLaunchKernelGGL(gemm_a2a_kernel, dim3(NWG), dim3(NTHREADS), LDS_BYTES, stream, P);
    HIP_CHECK(hipGetLastError());
}
```

```cpp
#include <hip/hip_runtime.h>
#include <rccl/rccl.h>
#include <cstdio>
#include <cstdint>

#define HIP_CHECK(expr) do { hipError_t _e = (expr); if (_e != hipSuccess) { \
    fprintf(stderr, "HIP error %s at %s:%d\n", hipGetErrorString(_e), __FILE__, __LINE__); \
    return; } } while (0)

typedef _Float16 f16x8 __attribute__((ext_vector_type(8)));
typedef _Float16 f16x2 __attribute__((ext_vector_type(2)));
typedef float f32x4 __attribute__((ext_vector_type(4)));
typedef float f32x2 __attribute__((ext_vector_type(2)));
typedef unsigned u32x4 __attribute__((ext_vector_type(4)));
typedef unsigned u32x2 __attribute__((ext_vector_type(2)));
typedef unsigned u32x8 __attribute__((ext_vector_type(8)));
typedef __attribute__((address_space(3))) volatile unsigned lds_vu32;

constexpr int NRANK = 8;
constexpr int ML = 256;
constexpr int KD = 2048;
constexpr int ND = 2048;
constexpr int NB = 256;
constexpr int NWG = 256;
constexpr int WG_PER_DST = NWG / NRANK;
constexpr int NTHREADS = 640;

constexpr unsigned WS_SEQ = 0;
constexpr unsigned WS_FLAG = 4096;
constexpr unsigned WS_TMO = 8192;
constexpr unsigned WS_X16 = 1u << 20;
constexpr unsigned WS_BYTES_USED = 3u << 20;
constexpr unsigned long long DEADLINE_TICKS = 50000000ull;
constexpr unsigned long long SELF_SERVE_TICKS = 3000ull;

#define LOCALG 4
struct __attribute__((aligned(128))) GroupParams {
    const char* wb[8];
    char* out;
    unsigned* flag;
    const float* x;
    unsigned* ws;
    unsigned ptab;
    unsigned rank;
    unsigned long long* dbg;
    unsigned pad[4];
};
struct Params { GroupParams g[NRANK]; };

constexpr int AUX_SYS = 17;
constexpr int AUX_DEV = 16;

__device__ __forceinline__ unsigned cvt2(float a, float b) {
    f32x2 v = {a, b};
    return __builtin_bit_cast(unsigned, __builtin_convertvector(v, f16x2));
}
__device__ __forceinline__ unsigned umin3(unsigned a, unsigned b, unsigned c) {
    return min(min(a, b), c);
}
#define SB() __builtin_amdgcn_sched_barrier(0)
#define BARRIER() do { SB(); asm volatile("s_barrier" ::: "memory"); SB(); } while (0)
#define STAMP(i)
#define PME 0x77777777u
#define PMO 0x55557777u
#define PML 4u
#define HT 0x012233u
#define NH 6
#define NFULL 4
#define CVO 1
#define DEP 4
#define WROLL 1
#define OFFT 0x76543210u
#define SXT 0x00000000u
#define SLOOP 0
#define LEXP 0
#define ONEBAR 1
#define WDEP DEP

constexpr int LDS_RED = 4 * 8 * 2048;
constexpr int LDS_PAR = LDS_RED;
constexpr int LDS_CNT = LDS_RED + 64;
constexpr int LDS_BYTES = LDS_RED + 512;

__global__ void __launch_bounds__(NTHREADS) gemm_a2a_kernel(Params P) {
    extern __shared__ __attribute__((aligned(16))) char smem[];
    const int tid = threadIdx.x;
    const int lane = tid & 63;
    const int wave = __builtin_amdgcn_readfirstlane(tid >> 6) - 2;
    const int b = blockIdx.x;
    const int grp = b & 7, idx = b >> 3;
    const int mt = idx & 3, nv = idx >> 2;
    const int m0 = mt * 64;
    const unsigned sx = (SXT >> (4 * nv)) & 3u;
    STAMP(0);
    const char* ka = (const char*)__builtin_amdgcn_kernarg_segment_ptr() + grp * 128;
    asm volatile("s_load_dword s100, %0, 0x0\n\ts_load_dword s101, %0, 0x40" :: "s"(ka) : "s100", "s101", "memory");
    SB();
#define PTR(lo, hi) ((void*)(((unsigned long long)(hi) << 32) | (lo)))
#define RSRC(lo, hi, bytes) __builtin_amdgcn_make_buffer_rsrc(PTR(lo, hi), 0, bytes, 0x00020000)
    constexpr unsigned W_SPAN = (KD - 1) * ND * 4 + 128;

    int lane8 = lane >> 3, lane16 = lane * 16, wv2 = wave;
    asm volatile("" : "+v"(lane8), "+v"(lane16), "+s"(wv2));
    if (__builtin_expect(wave < 0, 0)) {
    const int pw = wave + 2;
    __builtin_amdgcn_s_setprio(3);
    u32x2 misc;
    u32x8 ptrs;
    asm volatile("s_load_dwordx8 %0, %2, 0x40\n\ts_load_dwordx2 %1, %2, 0x60\n\ts_waitcnt lgkmcnt(0)" : "=&s"(ptrs), "=&s"(misc) : "s"(ka) : "memory");
    const __amdgpu_buffer_rsrc_t sr = RSRC(ptrs[6], ptrs[7], WS_BYTES_USED);
    const unsigned callv = __builtin_amdgcn_raw_buffer_load_b32(sr, 0, WS_SEQ + b * 4, AUX_DEV);
    if (!ONEBAR && (nv >= 4 || grp == LOCALG)) BARRIER();
    STAMP(10);
    const unsigned call = __builtin_amdgcn_readfirstlane(callv) + 1u;
    if (pw == 0 && lane == 0) *(unsigned*)(smem + LDS_PAR) = call & 1u;
    if (pw == 1 && lane < 8) *(unsigned*)(smem + LDS_CNT + lane * 4) = 0u;
    asm volatile("s_waitcnt lgkmcnt(0)" ::: "memory");
    if (ONEBAR && SLOOP) BARRIER();
    STAMP(1);
    const unsigned rank = misc[1] & 7u;
    const unsigned nt = (misc[0] >> (4 * nv)) & 7u;
    const __amdgpu_buffer_rsrc_t orr = __builtin_amdgcn_make_buffer_rsrc(PTR(ptrs[0], ptrs[1]), 0, ML * NB * 4, 0x00020000);
    const __amdgpu_buffer_rsrc_t fr = __builtin_amdgcn_make_buffer_rsrc(PTR(ptrs[2], ptrs[3]), 0, WG_PER_DST * 4, 0x00020000);
    const unsigned ooff = (unsigned)((m0 + pw * 8 + (lane >> 3)) * NB + nt * 32 + (lane & 7) * 4) * 4u;
    const unsigned pm = grp == LOCALG ? PML : ((mt & 1 ? PMO : PME) >> (4 * nv)) & 7u;
    asm volatile("" :: "v"(ooff), "s"(pm));
#pragma unroll
    for (int g = 0; g < (WROLL ? 1 : 7); ++g) BARRIER();
    bool timed_out = false;
    f32x4 sv[4];
#pragma unroll
    for (int st = 0; st < 4; ++st) {
        for (unsigned n = 1, t0 = 0;; ++n) {
            const unsigned c = *(lds_vu32*)(smem + LDS_CNT + (lane & 7) * 4);
            if (__all(c > (unsigned)st)) break;
            if (__builtin_expect((n & 4095u) == 0u, 0)) {
                const unsigned t = (unsigned)__builtin_amdgcn_s_memrealtime();
                if (n == 4096u) t0 = t;
                if (t - t0 > (unsigned)DEADLINE_TICKS) { timed_out = true; break; }
            }
        }
        asm volatile("" ::: "memory");
        SB();
        const char* red = smem + st * 16384 + pw * 1024 + lane * 16;
        f32x4 v[8];
#pragma unroll
        for (int w = 0; w < 8; ++w) v[w] = *(const f32x4*)(red + w * 2048);
        SB();
        f32x4 s = ((v[0] + v[1]) + (v[2] + v[3])) + ((v[4] + v[5]) + (v[6] + v[7]));
        s[0] = fmaxf(s[0], 0.f); s[1] = fmaxf(s[1], 0.f); s[2] = fmaxf(s[2], 0.f); s[3] = fmaxf(s[3], 0.f);
        sv[st] = s;
        if (__builtin_expect(timed_out, 0)) sv[st][0] = __builtin_nanf("");
#define PUSH(j) __builtin_amdgcn_raw_buffer_store_b128(__builtin_bit_cast(u32x4, sv[j]), orr, ooff, ((j) ^ sx) * (16 * NB * 4), AUX_SYS)
        if (st == 0) { if (pm & 1u) PUSH(0); }
        if (st == 1) { if (pm & 2u) { if (!(pm & 1u)) PUSH(0); PUSH(1); } }
        if (st == 2) { if (pm & 4u) { if (!(pm & 3u)) PUSH(0); if (!(pm & 2u)) PUSH(1); PUSH(2); } }
        if (st == 3) { if (!pm) PUSH(0); if (!(pm & 6u)) PUSH(1); if (!(pm & 4u)) PUSH(2); PUSH(3); }
        STAMP(3 + st);
    }
    {
        const u32x4 z = {0u, 0u, 0u, 0u};
        const unsigned so = WS_X16 + (((call & 1u) ^ 1u) << 20) + (unsigned)(b * 2 + pw) * 2048u;
        __builtin_amdgcn_raw_buffer_store_b128(z, sr, lane * 16, so, AUX_DEV);
        __builtin_amdgcn_raw_buffer_store_b128(z, sr, lane * 16 + 1024, so, AUX_DEV);
        asm volatile("s_waitcnt vmcnt(2)\n\ts_barrier" ::: "memory");
    }
    STAMP(7);
    if (__builtin_expect(timed_out, 0) && lane == 0) __builtin_amdgcn_raw_buffer_store_b32(call, sr, 0, WS_TMO, AUX_DEV);
    if (pw == 0) {
        if (lane == 0) {
            if (grp != LOCALG) __builtin_amdgcn_raw_buffer_store_b32(call, fr, 0, idx * 4, AUX_SYS);
            __builtin_amdgcn_raw_buffer_store_b32(call, sr, 0, WS_SEQ + b * 4, AUX_DEV);
        }
        if (b == LOCALG) {
            const bool self = (unsigned)(lane >> 3) == rank;
            const unsigned long long tw = __builtin_amdgcn_s_memrealtime();
            for (;;) {
                asm volatile("" ::: "memory");
                u32x4 f = __builtin_amdgcn_raw_buffer_load_b128(sr, lane * 16, WS_FLAG, AUX_SYS);
                bool ok = self || (f[0] >= call && f[1] >= call && f[2] >= call && f[3] >= call);
                if (__all(ok)) break;
                if (__builtin_amdgcn_s_memrealtime() - tw > DEADLINE_TICKS) { timed_out = true; break; }
                __builtin_amdgcn_s_sleep(1);
            }
            if (timed_out && lane == 0) {
                __builtin_amdgcn_raw_buffer_store_b32(call, sr, 0, WS_TMO, AUX_DEV);
                __builtin_amdgcn_raw_buffer_store_b32(0x7fc00000u, orr, 0, nt * 128, AUX_SYS);
            }
        }
    }
    asm volatile("s_endpgm" ::: "memory");
    __builtin_amdgcn_s_waitcnt(0);
    }

    {
        if (__builtin_expect(grp == LOCALG, LEXP)) {
#define CLOAD(u, rb) _Pragma("unroll") for (int i = 0; i < 4; ++i) \
            px[4 * (u) + i] = __builtin_bit_cast(f32x4, __builtin_amdgcn_raw_buffer_load_b128(xr, cvo + (i >> 1) * 128, ((i & 1) * 8 + (rb) * 16) * KD * 4, 0))
#define CONVERT(u, rb) do { \
            const unsigned so = WS_X16 + (((__builtin_amdgcn_readfirstlane(callv) + 1u) & 1u) << 20) + (unsigned)((crb + (rb)) * 64 + cks) * 1024u; \
            const int c = lane & 7; \
            char* tb = smem + wave * 4096 + (u) * 2048; \
            _Pragma("unroll") for (int i = 0; i < 4; ++i) { \
                u32x2 h = {cvt2(px[4 * (u) + i][0], px[4 * (u) + i][1]), cvt2(px[4 * (u) + i][2], px[4 * (u) + i][3])}; \
                *(u32x2*)(tb + (i >> 1) * 1024 + ((i & 1) * 8 + (lane >> 3) + 16 * (c >> 1)) * 16 + (c & 1) * 8) = h; \
            } \
            asm volatile("s_waitcnt lgkmcnt(0)" ::: "memory"); \
            _Pragma("unroll") for (int j = 0; j < 2; ++j) { \
                u32x4 h = *(const u32x4*)(tb + j * 1024 + lane * 16); \
                _Pragma("unroll") for (int i = 0; i < 4; ++i) h[i] = h[i] == 0u ? 0x8000u : h[i]; \
                asm volatile("buffer_store_dwordx4 %0, %1, %2, %3 offen sc1\n\ts_nop 1" :: "v"(h), "v"(lane * 16 + j * 1024), "s"(sr), "s"(so) : "memory"); \
            } \
            SB(); } while (0)
            u32x4 xw;
            asm volatile("s_load_dwordx4 %0, %1, 0x50" : "=&s"(xw) : "s"(ka) : "memory");
            const int crb = (idx >> 3) * 4 + (wave >> 2), cks = ((idx & 7) * 4 + (wave & 3)) * 2;
            const unsigned cvo = (unsigned)((crb * 16 + (lane >> 3)) * KD + cks * 32 + (lane & 7) * 4) * 4u;
            asm volatile("s_waitcnt lgkmcnt(0)" : "+s"(xw) : "v"(cvo) : "memory");
            const __amdgpu_buffer_rsrc_t xr = RSRC(xw[0], xw[1], ML * KD * 4), sr = RSRC(xw[2], xw[3], WS_BYTES_USED);
            f32x4 px[8];
            const unsigned callv = __builtin_amdgcn_raw_buffer_load_b32(sr, 0, WS_SEQ + b * 4, AUX_DEV);
            CLOAD(0, 0);
            SB();
            CONVERT(0, 0);
            CLOAD(1, 2);
            SB();
            STAMP(13);
            CONVERT(1, 2);
            STAMP(14);
        }
        u32x2 wown, whelp;
        u32x4 xw;
        const int ht = (HT >> (4 * wave)) & 3;
        asm volatile("s_load_dwordx2 %0, %3, %4\n\ts_load_dwordx2 %1, %3, %5\n\ts_load_dwordx4 %2, %3, 0x50"
                     : "=&s"(wown), "=&s"(whelp), "=&s"(xw) : "s"(ka), "s"(nv * 8), "s"(ht * 8) : "memory");
        const int kw = wave * 256;
        const int rot = (2 * mt + (OFFT >> (4 * nv))) & 7;
        const unsigned woff = (unsigned)((kw + 4 * (lane >> 3)) * ND + 4 * (lane & 7)) * 4u;
        const unsigned wso0 = (unsigned)rot * (32 * ND * 4);
        const unsigned hvo = (unsigned)(((lane >> 3) * 256 + (lane & 7)) * ND) * 4u;
        const unsigned hr0 = (idx & 15) * 16 + (wave & 1) * 8, hr1 = ((2 * nv + (OFFT >> (4 * ht)) + 1) & 7) * 32 + mt * 8;
        unsigned hso = (wave < NFULL ? hr0 : hr1) * (ND * 4u);
        asm volatile("s_waitcnt lgkmcnt(0)" : "+s"(wown), "+s"(whelp), "+s"(xw) : "v"(woff), "s"(wso0), "v"(hvo), "s"(hso), "s"(hr0), "s"(hr1) : "memory");
        const __amdgpu_buffer_rsrc_t wr = RSRC(wown[0], wown[1], W_SPAN);
        unsigned pf = 0;
        if (nv >= 4 || grp == LOCALG) {
            if (nv >= 4 && wave < NH) pf = __builtin_amdgcn_raw_buffer_load_b32(RSRC(whelp[0], whelp[1], W_SPAN), hvo, hso, 0);
            BARRIER();
        }
        const __amdgpu_buffer_rsrc_t xr = RSRC(xw[0], xw[1], ML * KD * 4), sr = RSRC(xw[2], xw[3], WS_BYTES_USED);
        const int li = lane & 15, lg = lane >> 4;
        STAMP(25);

        f32x4 wv[8][4];
        u32x4 wf[8][2];
        u32x4 xf[32];
#define LDW1(i) wv[(i) >> 2][(i) & 3] = __builtin_bit_cast(f32x4, __builtin_amdgcn_raw_buffer_load_b128(wr, woff, (((((i) >> 2) + rot) & 7) * 32 + ((i) & 3)) * ND * 4, 0))
#define LDWG(s) do { LDW1(4 * (s)); LDW1(4 * (s) + 1); LDW1(4 * (s) + 2); LDW1(4 * (s) + 3); } while (0)
#define CVW(s) _Pragma("unroll") for (int t = 0; t < 2; ++t) _Pragma("unroll") for (int d = 0; d < 2; ++d) { \
        unsigned p0 = cvt2(wv[s][2 * d][t], wv[s][2 * d + 1][t]); \
        unsigned p2 = cvt2(wv[s][2 * d][2 + t], wv[s][2 * d + 1][2 + t]); \
        wf[s][t][d] = __builtin_amdgcn_update_dpp(p0, p2, 0x128, 0xf, 0xc, false); \
        wf[s][t][2 + d] = __builtin_amdgcn_update_dpp(p2, p0, 0x128, 0xf, 0x3, false); }
#define XLD(q) __builtin_amdgcn_raw_buffer_load_b128(sr, hoff, (hbase ^ (((q) >> 3) * 65536)) + ((((q) & 7) + rot) & 7) * 1024, AUX_DEV)
#define LDX1(q) xf[q] = XLD(q)
#define LDX4(q) do { LDX1(q); LDX1((q) + 1); LDX1((q) + 2); LDX1((q) + 3); } while (0)
#define WAITV(v) do { SB(); asm volatile("" :: "v"(v)); SB(); } while (0)
#define WAITW(s) WAITV(wv[s][3])
        const unsigned hoff = (unsigned)(lane * 16 + wave * 8192);
#pragma unroll
        for (int i = 0; i < WDEP; ++i) { LDW1(i); SB(); }
        if (!ONEBAR || !(nv >= 4 || grp == LOCALG)) BARRIER();
        const unsigned par = __builtin_amdgcn_readfirstlane(*(const unsigned*)(smem + LDS_PAR));
        const unsigned hbase = WS_X16 + (par << 20) + (unsigned)mt * 262144u + sx * 65536u;
        asm volatile("" :: "v"(pf), "s"(hbase));
#pragma unroll
        for (int i = 0; i < 32; ++i) {
            WAITV(wv[i >> 2][i & 3]);
            if (i + WDEP < 32) LDW1((i + WDEP) & 31); else LDX1((i + WDEP) & 31);
            SB();
            if ((i & 3) == 3) { STAMP(17 + (i >> 2)); CVW(i >> 2); }
        }
        SB();
        constexpr int XPRE = WDEP;
        STAMP(2);

#define CNT_RETRY
#define MMA(f, s) do { \
            f16x8 af = __builtin_bit_cast(f16x8, f); \
            acc0 = __builtin_amdgcn_mfma_f32_16x16x32_f16(af, __builtin_bit_cast(f16x8, wf[s][0]), acc0, 0, 0, 0); \
            acc1 = __builtin_amdgcn_mfma_f32_16x16x32_f16(af, __builtin_bit_cast(f16x8, wf[s][1]), acc1, 0, 0, 0); } while (0)
#pragma unroll
        for (int st = 0; st < 4; ++st) {
            f32x4 acc0 = {0.f, 0.f, 0.f, 0.f}, acc1 = {0.f, 0.f, 0.f, 0.f};
            unsigned m = 0xffffffffu;
#pragma unroll
            for (int s = 0; s < 8; ++s) {
                const int q = st * 8 + s;
                WAITV(xf[q]);
                if (q + DEP >= XPRE && q + DEP < 32) LDX1((q + DEP) & 31);
                if (q == 0) { _Pragma("unroll") for (int e = XPRE; e < DEP; ++e) LDX1(e); }
                SB();
                MMA(xf[q], s);
                m = umin3(m, xf[q][0], xf[q][1]); m = umin3(m, xf[q][2], xf[q][3]);
            }
            if (st == 0) STAMP(12);
            if (__builtin_expect(__any(m == 0u), 0)) {
                const unsigned long long tw = __builtin_amdgcn_s_memrealtime();
                u32x4 f[8];
                for (;;) {
                    CNT_RETRY;
                    asm volatile("" ::: "memory");
                    m = 0xffffffffu;
#pragma unroll
                    for (int s = 0; s < 8; ++s) f[s] = XLD(st * 8 + s);
#pragma unroll
                    for (int s = 0; s < 8; ++s) { m = umin3(m, f[s][0], f[s][1]); m = umin3(m, f[s][2], f[s][3]); }
                    if (!__any(m == 0u)) break;
                    if (__builtin_amdgcn_s_memrealtime() - tw > SELF_SERVE_TICKS) {
                        const unsigned fo = (unsigned)((m0 + (st ^ sx) * 16 + li) * KD + kw + lg * 8) * 4u;
#pragma unroll
                        for (int s = 0; s < 8; ++s) {
                            f32x4 qa = __builtin_bit_cast(f32x4, __builtin_amdgcn_raw_buffer_load_b128(xr, fo, ((s + rot) & 7) * 128, 0));
                            f32x4 qb = __builtin_bit_cast(f32x4, __builtin_amdgcn_raw_buffer_load_b128(xr, fo + 16, ((s + rot) & 7) * 128, 0));
                            f[s] = u32x4{cvt2(qa[0], qa[1]), cvt2(qa[2], qa[3]), cvt2(qb[0], qb[1]), cvt2(qb[2], qb[3])};
                            SB();
                        }
                        break;
                    }
                }
                acc0 = f32x4{0.f, 0.f, 0.f, 0.f}; acc1 = f32x4{0.f, 0.f, 0.f, 0.f};
#pragma unroll
                for (int s = 0; s < 8; ++s) MMA(f[s], s);
            }
            char* red = smem + st * 16384;
#pragma unroll
            for (int r = 0; r < 4; ++r) {
                f32x2 v = {acc0[r], acc1[r]};
                *(f32x2*)(red + wave * 2048 + (lg * 4 + r) * 128 + (lane & 7) * 16 + ((lane >> 3) & 1) * 8) = v;
            }
            SB();
            asm volatile("" ::: "memory");
            *(lds_vu32*)(smem + LDS_CNT + wave * 4) = st + 1;
            SB();
            if (st == 0) STAMP(11);
        }
    }
}

extern "C" void kernel_launch(int rank, int nranks, ncclComm_t comm,
                              hipStream_t stream,
                              void* const* d_in, const int* in_sizes, int n_in,
                              void* d_out, int out_size,
                              const int* in_dtypes, int out_dtype,
                              void* const* d_out_all,
                              void* const* d_ws_all, size_t ws_bytes) {
    if (nranks != NRANK || (unsigned)rank >= (unsigned)NRANK || ws_bytes < WS_BYTES_USED) {
        fprintf(stderr, "unexpected configuration\n");
        return;
    }
    Params P = {};
    const char* w = (const char*)d_in[1];
    const unsigned wsh = (0u - (unsigned)((uintptr_t)w >> 7)) & 7u;
    for (int g = 0; g < NRANK; ++g) {
        const int dst = (rank + g - LOCALG) & 7;
        GroupParams& G = P.g[g];
        for (int v = 0; v < 8; ++v) {
            const unsigned phys = ((0x26543710u >> (4 * v)) + wsh) & 7u;
            G.wb[v] = w + dst * (NB * 4) + phys * 128;
            G.ptab |= phys << (4 * v);
        }
        G.out = (char*)d_out_all[dst] + (size_t)rank * ML * NB * 4;
        G.flag = (unsigned*)d_ws_all[dst] + WS_FLAG / 4 + rank * WG_PER_DST;
        G.x = (const float*)d_in[0];
        G.ws = (unsigned*)d_ws_all[rank];
        G.rank = (unsigned)rank;
        G.dbg = (unsigned long long*)((char*)d_ws_all[rank] + 65536);
    }
    HIP_CHECK(hipFuncSetAttribute((const void*)gemm_a2a_kernel, hipFuncAttributeMaxDynamicSharedMemorySize, LDS_BYTES));
    hipLaunchKernelGGL(gemm_a2a_kernel, dim3(NWG), dim3(NTHREADS), LDS_BYTES, stream, P);
    HIP_CHECK(hipGetLastError());
}
```

```cpp
#include <hip/hip_runtime.h>
#include <rccl/rccl.h>
#include <cstdio>
#include <cstdint>

#define HIP_CHECK(expr) do { hipError_t _e = (expr); if (_e != hipSuccess) { \
    fprintf(stderr, "HIP error %s at %s:%d\n", hipGetErrorString(_e), __FILE__, __LINE__); \
    return; } } while (0)

typedef _Float16 f16x8 __attribute__((ext_vector_type(8)));
typedef _Float16 f16x2 __attribute__((ext_vector_type(2)));
typedef float f32x4 __attribute__((ext_vector_type(4)));
typedef float f32x2 __attribute__((ext_vector_type(2)));
typedef unsigned u32x4 __attribute__((ext_vector_type(4)));
typedef unsigned u32x2 __attribute__((ext_vector_type(2)));
typedef unsigned u32x8 __attribute__((ext_vector_type(8)));
typedef __attribute__((address_space(3))) volatile unsigned lds_vu32;

constexpr int NRANK = 8;
constexpr int ML = 256;
constexpr int KD = 2048;
constexpr int ND = 2048;
constexpr int NB = 256;
constexpr int NWG = 256;
constexpr int WG_PER_DST = NWG / NRANK;
constexpr int NTHREADS = 640;

constexpr unsigned WS_SEQ = 0;
constexpr unsigned WS_FLAG = 4096;
constexpr unsigned WS_PROBE = 5120;
constexpr unsigned FLAG_COPY = 8192;
constexpr unsigned WS_TMO = 8192;
constexpr unsigned WS_X16 = 1u << 20;
constexpr unsigned WS_BYTES_USED = 3u << 20;
constexpr unsigned long long DEADLINE_TICKS = 50000000ull;
constexpr unsigned long long SELF_SERVE_TICKS = 3000ull;

#define LOCALG 4
struct __attribute__((aligned(128))) GroupParams {
    const char* wb[8];
    char* out;
    unsigned* flag;
    const float* x;
    unsigned* ws;
    unsigned ptab;
    unsigned rank;
    unsigned long long* dbg;
    char* lout;
    unsigned peer;
    unsigned pad;
};
struct Params { GroupParams g[NRANK]; };

constexpr int AUX_SYS = 17;
constexpr int AUX_DEV = 16;
constexpr int AUX_NT = 2;

__device__ __forceinline__ unsigned cvt2(float a, float b) {
    f32x2 v = {a, b};
    return __builtin_bit_cast(unsigned, __builtin_convertvector(v, f16x2));
}
__device__ __forceinline__ unsigned umin3(unsigned a, unsigned b, unsigned c) {
    return min(min(a, b), c);
}
#define SB() __builtin_amdgcn_sched_barrier(0)
#define BARRIER() do { SB(); asm volatile("s_barrier" ::: "memory"); SB(); } while (0)
#define STAMP(i)
#define PME 0x77777777u
#define PMO 0x11117777u
#define PML 4u
#define HT 0x012233u
#define NH 6
#define NFULL 4
#define CVO 1
#define DEP 4
#define WROLL 1
#define OFFT 0x76543210u
#define SXT 0x00000000u
#define SLOOP 0
#define LEXP 0
#define ONEBAR 1
#define PIDX 23
#define NPR 4
#define CLQ 0xB2u
#define SAMEM 0x0du
#define WDEP DEP

constexpr int LDS_RED = 4 * 8 * 2048;
constexpr int LDS_PAR = LDS_RED;
constexpr int LDS_CNT = LDS_RED + 64;
constexpr int LDS_BYTES = LDS_RED + 512;

__global__ void __launch_bounds__(NTHREADS) gemm_a2a_kernel(Params P) {
    extern __shared__ __attribute__((aligned(16))) char smem[];
    const int tid = threadIdx.x;
    const int lane = tid & 63;
    const int wave = __builtin_amdgcn_readfirstlane(tid >> 6) - 2;
    const int b = blockIdx.x;
    const int grp = b & 7, idx = b >> 3;
    const int mt = idx & 3, nv = idx >> 2;
    const int m0 = mt * 64;
    const unsigned sx = (SXT >> (4 * nv)) & 3u;
    STAMP(0);
    const char* ka = (const char*)__builtin_amdgcn_kernarg_segment_ptr() + grp * 128;
    asm volatile("s_load_dword s100, %0, 0x0\n\ts_load_dword s101, %0, 0x40" :: "s"(ka) : "s100", "s101", "memory");
    SB();
#define PTR(lo, hi) ((void*)(((unsigned long long)(hi) << 32) | (lo)))
#define RSRC(lo, hi, bytes) __builtin_amdgcn_make_buffer_rsrc(PTR(lo, hi), 0, bytes, 0x00020000)
    constexpr unsigned W_SPAN = (KD - 1) * ND * 4 + 128;

    int lane8 = lane >> 3, lane16 = lane * 16, wv2 = wave;
    asm volatile("" : "+v"(lane8), "+v"(lane16), "+s"(wv2));
    if (__builtin_expect(wave < 0, 0)) {
    const int pw = wave + 2;
    __builtin_amdgcn_s_setprio(3);
    u32x2 misc;
    u32x8 ptrs;
    asm volatile("s_load_dwordx8 %0, %2, 0x40\n\ts_load_dwordx2 %1, %2, 0x60\n\ts_waitcnt lgkmcnt(0)" : "=&s"(ptrs), "=&s"(misc) : "s"(ka) : "memory");
    const __amdgpu_buffer_rsrc_t sr = RSRC(ptrs[6], ptrs[7], WS_BYTES_USED);
    const unsigned callv = __builtin_amdgcn_raw_buffer_load_b32(sr, 0, WS_SEQ + b * 4, AUX_DEV);
    if (!ONEBAR && (nv >= 4 || grp == LOCALG)) BARRIER();
    STAMP(10);
    const unsigned call = __builtin_amdgcn_readfirstlane(callv) + 1u;
    if (pw == 0 && lane == 0) *(unsigned*)(smem + LDS_PAR) = call & 1u;
    if (pw == 1 && lane < 8) *(unsigned*)(smem + LDS_CNT + lane * 4) = 0u;
    asm volatile("s_waitcnt lgkmcnt(0)" ::: "memory");
    if (ONEBAR && SLOOP) BARRIER();
    STAMP(1);
    const unsigned nt = (misc[0] >> (4 * nv)) & 7u;
    const __amdgpu_buffer_rsrc_t orr = __builtin_amdgcn_make_buffer_rsrc(PTR(ptrs[0], ptrs[1]), 0, ML * NB * 4, 0x00020000);
    const __amdgpu_buffer_rsrc_t fr = __builtin_amdgcn_make_buffer_rsrc(PTR(ptrs[2], ptrs[3]), 0, FLAG_COPY + WG_PER_DST * 4, 0x00020000);
    const unsigned ooff = (unsigned)((m0 + pw * 8 + (lane >> 3)) * NB + nt * 32 + (lane & 7) * 4) * 4u;
    const unsigned pm = grp == LOCALG ? PML : ((mt & 1 ? PMO : PME) >> (4 * nv)) & 7u;
    asm volatile("" :: "v"(ooff), "s"(pm));
#pragma unroll
    for (int g = 0; g < (WROLL ? 1 : 7); ++g) BARRIER();
    const bool poller = pw == 1 && idx == PIDX && grp != LOCALG;
    unsigned fsel = 0;
    if (poller) {
        unsigned best[2] = {~0u, ~0u};
#pragma unroll 1
        for (int i = 0; i < NPR; ++i) {
#pragma unroll
            for (int c = 0; c < 2; ++c) {
                SB();
                const unsigned t0 = (unsigned)__builtin_amdgcn_s_memtime();
                asm volatile("s_waitcnt lgkmcnt(0)" ::: "memory");
                if (lane == 0) __builtin_amdgcn_raw_buffer_store_b32(call, sr, 0, WS_PROBE + c * FLAG_COPY + grp * 128, AUX_SYS);
                asm volatile("s_waitcnt vmcnt(0)" ::: "memory");
                const unsigned t1 = (unsigned)__builtin_amdgcn_s_memtime();
                SB();
                best[c] = min(best[c], t1 - t0);
            }
        }
        fsel = best[1] < best[0] ? FLAG_COPY : 0u;
    }
    bool timed_out = false;
    f32x4 sv[4];
#pragma unroll
    for (int st = 0; st < 4; ++st) {
        for (unsigned n = 1, t0 = 0;; ++n) {
            const unsigned c = *(lds_vu32*)(smem + LDS_CNT + (lane & 7) * 4);
            if (__all(c > (unsigned)st)) break;
            if (__builtin_expect((n & 4095u) == 0u, 0)) {
                const unsigned t = (unsigned)__builtin_amdgcn_s_memrealtime();
                if (n == 4096u) t0 = t;
                if (t - t0 > (unsigned)DEADLINE_TICKS) { timed_out = true; break; }
            }
        }
        asm volatile("" ::: "memory");
        SB();
        const char* red = smem + st * 16384 + pw * 1024 + lane * 16;
        f32x4 v[8];
#pragma unroll
        for (int w = 0; w < 8; ++w) v[w] = *(const f32x4*)(red + w * 2048);
        SB();
        f32x4 s = ((v[0] + v[1]) + (v[2] + v[3])) + ((v[4] + v[5]) + (v[6] + v[7]));
        s[0] = fmaxf(s[0], 0.f); s[1] = fmaxf(s[1], 0.f); s[2] = fmaxf(s[2], 0.f); s[3] = fmaxf(s[3], 0.f);
        sv[st] = s;
        if (__builtin_expect(timed_out, 0)) sv[st][0] = __builtin_nanf("");
#define PUSH(j) __builtin_amdgcn_raw_buffer_store_b128(__builtin_bit_cast(u32x4, sv[j]), orr, ooff, ((j) ^ sx) * (16 * NB * 4), AUX_SYS)
        if (st == 0) { if (pm & 1u) PUSH(0); }
        if (st == 1) { if (pm & 2u) { if (!(pm & 1u)) PUSH(0); PUSH(1); } }
        if (st == 2) { if (pm & 4u) { if (!(pm & 3u)) PUSH(0); if (!(pm & 2u)) PUSH(1); PUSH(2); } }
        if (st == 3) { if (!pm) PUSH(0); if (!(pm & 6u)) PUSH(1); if (!(pm & 4u)) PUSH(2); PUSH(3); }
        STAMP(3 + st);
    }
    {
        const u32x4 z = {0u, 0u, 0u, 0u};
        const unsigned so = WS_X16 + (((call & 1u) ^ 1u) << 20) + (unsigned)(b * 2 + pw) * 2048u;
        __builtin_amdgcn_raw_buffer_store_b128(z, sr, lane * 16, so, AUX_DEV);
        __builtin_amdgcn_raw_buffer_store_b128(z, sr, lane * 16 + 1024, so, AUX_DEV);
        asm volatile("s_waitcnt vmcnt(2)\n\ts_barrier" ::: "memory");
    }
    STAMP(7);
    if (__builtin_expect(timed_out, 0) && lane == 0) __builtin_amdgcn_raw_buffer_store_b32(call, sr, 0, WS_TMO, AUX_DEV);
    if (pw == 0) {
        if (lane < 2 && grp != LOCALG) __builtin_amdgcn_raw_buffer_store_b32(call, fr, lane * FLAG_COPY, idx * 4, AUX_SYS);
        if (lane == 0) __builtin_amdgcn_raw_buffer_store_b32(call, sr, 0, WS_SEQ + b * 4, AUX_DEV);
    }
    if (poller) {
        u32x4 lp;
        asm volatile("s_load_dwordx4 %0, %1, 0x70\n\ts_waitcnt lgkmcnt(0)" : "=&s"(lp) : "s"(ka) : "memory");
        const unsigned peer = lp[2] & 7u;
        const unsigned long long tw = __builtin_amdgcn_s_memrealtime();
        for (;;) {
            asm volatile("" ::: "memory");
            const unsigned f = __builtin_amdgcn_raw_buffer_load_b32(sr, (lane & 31) * 4, WS_FLAG + fsel + peer * 128u, AUX_SYS);
            if (__all(f >= call)) break;
            if (__builtin_amdgcn_s_memrealtime() - tw > DEADLINE_TICKS) { timed_out = true; break; }
        }
        if (timed_out && lane == 0) {
            __builtin_amdgcn_raw_buffer_store_b32(call, sr, 0, WS_TMO, AUX_DEV);
            __builtin_amdgcn_raw_buffer_store_b32(0x7fc00000u, RSRC(lp[0], lp[1], NRANK * ML * NB * 4), 0, peer * (ML * NB * 4), AUX_SYS);
        }
    }
    asm volatile("s_endpgm" ::: "memory");
    __builtin_amdgcn_s_waitcnt(0);
    }

    {
        if (__builtin_expect(grp == LOCALG, LEXP)) {
#define CLOAD(u, rb) _Pragma("unroll") for (int i = 0; i < 4; ++i) \
            px[4 * (u) + i] = __builtin_bit_cast(f32x4, __builtin_amdgcn_raw_buffer_load_b128(xr, cvo + (i >> 1) * 128, ((i & 1) * 8 + (rb) * 16) * KD * 4, AUX_NT))
#define CONVERT(u, rb) do { \
            const unsigned so = WS_X16 + (((__builtin_amdgcn_readfirstlane(callv) + 1u) & 1u) << 20) + (unsigned)((crb + (rb)) * 64 + cks) * 1024u; \
            const int c = lane & 7; \
            char* tb = smem + wave * 4096 + (u) * 2048; \
            _Pragma("unroll") for (int i = 0; i < 4; ++i) { \
                u32x2 h = {cvt2(px[4 * (u) + i][0], px[4 * (u) + i][1]), cvt2(px[4 * (u) + i][2], px[4 * (u) + i][3])}; \
                *(u32x2*)(tb + (i >> 1) * 1024 + ((i & 1) * 8 + (lane >> 3) + 16 * (c >> 1)) * 16 + (c & 1) * 8) = h; \
            } \
            asm volatile("s_waitcnt lgkmcnt(0)" ::: "memory"); \
            _Pragma("unroll") for (int j = 0; j < 2; ++j) { \
                u32x4 h = *(const u32x4*)(tb + j * 1024 + lane * 16); \
                _Pragma("unroll") for (int i = 0; i < 4; ++i) h[i] = h[i] == 0u ? 0x8000u : h[i]; \
                asm volatile("buffer_store_dwordx4 %0, %1, %2, %3 offen sc1\n\ts_nop 1" :: "v"(h), "v"(lane * 16 + j * 1024), "s"(sr), "s"(so) : "memory"); \
            } \
            SB(); } while (0)
            u32x4 xw;
            asm volatile("s_load_dwordx4 %0, %1, 0x50" : "=&s"(xw) : "s"(ka) : "memory");
            const int crb = (idx >> 3) * 4 + (wave >> 2), cks = ((idx & 7) * 4 + (wave & 3)) * 2;
            const unsigned cvo = (unsigned)((crb * 16 + (lane >> 3)) * KD + cks * 32 + (lane & 7) * 4) * 4u;
            asm volatile("s_waitcnt lgkmcnt(0)" : "+s"(xw) : "v"(cvo) : "memory");
            const __amdgpu_buffer_rsrc_t xr = RSRC(xw[0], xw[1], ML * KD * 4), sr = RSRC(xw[2], xw[3], WS_BYTES_USED);
            f32x4 px[8];
            const unsigned callv = __builtin_amdgcn_raw_buffer_load_b32(sr, 0, WS_SEQ + b * 4, AUX_DEV);
            CLOAD(0, 0);
            SB();
            CONVERT(0, 0);
            CLOAD(1, 2);
            SB();
            STAMP(13);
            CONVERT(1, 2);
            STAMP(14);
        }
        u32x2 wown, whelp;
        u32x4 xw;
        const int ht = (HT >> (4 * wave)) & 3;
        asm volatile("s_load_dwordx2 %0, %3, %4\n\ts_load_dwordx2 %1, %3, %5\n\ts_load_dwordx4 %2, %3, 0x50"
                     : "=&s"(wown), "=&s"(whelp), "=&s"(xw) : "s"(ka), "s"(nv * 8), "s"(ht * 8) : "memory");
        const int kw = wave * 256;
        const int rot = (2 * mt + (OFFT >> (4 * nv))) & 7;
        const unsigned woff = (unsigned)((kw + 4 * (lane >> 3)) * ND + 4 * (lane & 7)) * 4u;
        const unsigned wso0 = (unsigned)rot * (32 * ND * 4);
        const unsigned hvo = (unsigned)(((lane >> 3) * 256 + (lane & 7)) * ND) * 4u;
        const unsigned hr0 = (idx & 15) * 16 + (wave & 1) * 8, hr1 = ((2 * nv + (OFFT >> (4 * ht)) + 1) & 7) * 32 + mt * 8;
        unsigned hso = (wave < NFULL ? hr0 : hr1) * (ND * 4u);
        asm volatile("s_waitcnt lgkmcnt(0)" : "+s"(wown), "+s"(whelp), "+s"(xw) : "v"(woff), "s"(wso0), "v"(hvo), "s"(hso), "s"(hr0), "s"(hr1) : "memory");
        const __amdgpu_buffer_rsrc_t wr = RSRC(wown[0], wown[1], W_SPAN);
        unsigned pf = 0;
        if (nv >= 4 || grp == LOCALG) {
            if (nv >= 4 && wave < NH) pf = __builtin_amdgcn_raw_buffer_load_b32(RSRC(whelp[0], whelp[1], W_SPAN), hvo, hso, 0);
            BARRIER();
        }
        const __amdgpu_buffer_rsrc_t xr = RSRC(xw[0], xw[1], ML * KD * 4), sr = RSRC(xw[2], xw[3], WS_BYTES_USED);
        const int li = lane & 15, lg = lane >> 4;
        STAMP(25);

        f32x4 wv[8][4];
        u32x4 wf[8][2];
        u32x4 xf[32];
#define LDW1(i) wv[(i) >> 2][(i) & 3] = __builtin_bit_cast(f32x4, __builtin_amdgcn_raw_buffer_load_b128(wr, woff, (((((i) >> 2) + rot) & 7) * 32 + ((i) & 3)) * ND * 4, 0))
#define LDWG(s) do { LDW1(4 * (s)); LDW1(4 * (s) + 1); LDW1(4 * (s) + 2); LDW1(4 * (s) + 3); } while (0)
#define CVW(s) _Pragma("unroll") for (int t = 0; t < 2; ++t) _Pragma("unroll") for (int d = 0; d < 2; ++d) { \
        unsigned p0 = cvt2(wv[s][2 * d][t], wv[s][2 * d + 1][t]); \
        unsigned p2 = cvt2(wv[s][2 * d][2 + t], wv[s][2 * d + 1][2 + t]); \
        wf[s][t][d] = __builtin_amdgcn_update_dpp(p0, p2, 0x128, 0xf, 0xc, false); \
        wf[s][t][2 + d] = __builtin_amdgcn_update_dpp(p2, p0, 0x128, 0xf, 0x3, false); }
#define XLD(q) __builtin_amdgcn_raw_buffer_load_b128(sr, hoff, (hbase ^ (((q) >> 3) * 65536)) + ((((q) & 7) + rot) & 7) * 1024, AUX_DEV)
#define LDX1(q) xf[q] = XLD(q)
#define LDX4(q) do { LDX1(q); LDX1((q) + 1); LDX1((q) + 2); LDX1((q) + 3); } while (0)
#define WAITV(v) do { SB(); asm volatile("" :: "v"(v)); SB(); } while (0)
#define WAITW(s) WAITV(wv[s][3])
        const unsigned hoff = (unsigned)(lane * 16 + wave * 8192);
#pragma unroll
        for (int i = 0; i < WDEP; ++i) { LDW1(i); SB(); }
        if (!ONEBAR || !(nv >= 4 || grp == LOCALG)) BARRIER();
        const unsigned par = __builtin_amdgcn_readfirstlane(*(const unsigned*)(smem + LDS_PAR));
        const unsigned hbase = WS_X16 + (par << 20) + (unsigned)mt * 262144u + sx * 65536u;
        asm volatile("" :: "v"(pf), "s"(hbase));
#pragma unroll
        for (int i = 0; i < 32; ++i) {
            WAITV(wv[i >> 2][i & 3]);
            if (i + WDEP < 32) LDW1((i + WDEP) & 31); else LDX1((i + WDEP) & 31);
            SB();
            if ((i & 3) == 3) { STAMP(17 + (i >> 2)); CVW(i >> 2); }
        }
        SB();
        constexpr int XPRE = WDEP;
        STAMP(2);

#define CNT_RETRY
#define MMA(f, s) do { \
            f16x8 af = __builtin_bit_cast(f16x8, f); \
            acc0 = __builtin_amdgcn_mfma_f32_16x16x32_f16(af, __builtin_bit_cast(f16x8, wf[s][0]), acc0, 0, 0, 0); \
            acc1 = __builtin_amdgcn_mfma_f32_16x16x32_f16(af, __builtin_bit_cast(f16x8, wf[s][1]), acc1, 0, 0, 0); } while (0)
#pragma unroll
        for (int st = 0; st < 4; ++st) {
            f32x4 acc0 = {0.f, 0.f, 0.f, 0.f}, acc1 = {0.f, 0.f, 0.f, 0.f};
            unsigned m = 0xffffffffu;
#pragma unroll
            for (int s = 0; s < 8; ++s) {
                const int q = st * 8 + s;
                WAITV(xf[q]);
                if (q + DEP >= XPRE && q + DEP < 32) LDX1((q + DEP) & 31);
                if (q == 0) { _Pragma("unroll") for (int e = XPRE; e < DEP; ++e) LDX1(e); }
                SB();
                MMA(xf[q], s);
                m = umin3(m, xf[q][0], xf[q][1]); m = umin3(m, xf[q][2], xf[q][3]);
            }
            if (st == 0) STAMP(12);
            if (__builtin_expect(__any(m == 0u), 0)) {
                const unsigned long long tw = __builtin_amdgcn_s_memrealtime();
                u32x4 f[8];
                for (;;) {
                    CNT_RETRY;
                    asm volatile("" ::: "memory");
                    m = 0xffffffffu;
#pragma unroll
                    for (int s = 0; s < 8; ++s) f[s] = XLD(st * 8 + s);
#pragma unroll
                    for (int s = 0; s < 8; ++s) { m = umin3(m, f[s][0], f[s][1]); m = umin3(m, f[s][2], f[s][3]); }
                    if (!__any(m == 0u)) break;
                    if (__builtin_amdgcn_s_memrealtime() - tw > SELF_SERVE_TICKS) {
                        const unsigned fo = (unsigned)((m0 + (st ^ sx) * 16 + li) * KD + kw + lg * 8) * 4u;
#pragma unroll
                        for (int s = 0; s < 8; ++s) {
                            f32x4 qa = __builtin_bit_cast(f32x4, __builtin_amdgcn_raw_buffer_load_b128(xr, fo, ((s + rot) & 7) * 128, 0));
                            f32x4 qb = __builtin_bit_cast(f32x4, __builtin_amdgcn_raw_buffer_load_b128(xr, fo + 16, ((s + rot) & 7) * 128, 0));
                            f[s] = u32x4{cvt2(qa[0], qa[1]), cvt2(qa[2], qa[3]), cvt2(qb[0], qb[1]), cvt2(qb[2], qb[3])};
                            SB();
                        }
                        break;
                    }
                }
                acc0 = f32x4{0.f, 0.f, 0.f, 0.f}; acc1 = f32x4{0.f, 0.f, 0.f, 0.f};
#pragma unroll
                for (int s = 0; s < 8; ++s) MMA(f[s], s);
            }
            char* red = smem + st * 16384;
#pragma unroll
            for (int r = 0; r < 4; ++r) {
                f32x2 v = {acc0[r], acc1[r]};
                *(f32x2*)(red + wave * 2048 + (lg * 4 + r) * 128 + (lane & 7) * 16 + ((lane >> 3) & 1) * 8) = v;
            }
            SB();
            asm volatile("" ::: "memory");
            *(lds_vu32*)(smem + LDS_CNT + wave * 4) = st + 1;
            SB();
            if (st == 0) STAMP(11);
        }
    }
}

extern "C" void kernel_launch(int rank, int nranks, ncclComm_t comm,
                              hipStream_t stream,
                              void* const* d_in, const int* in_sizes, int n_in,
                              void* d_out, int out_size,
                              const int* in_dtypes, int out_dtype,
                              void* const* d_out_all,
                              void* const* d_ws_all, size_t ws_bytes) {
    if (nranks != NRANK || (unsigned)rank >= (unsigned)NRANK || ws_bytes < WS_BYTES_USED) {
        fprintf(stderr, "unexpected configuration\n");
        return;
    }
    Params P = {};
    const char* w = (const char*)d_in[1];
    const unsigned wsh = (0u - (unsigned)((uintptr_t)w >> 7)) & 7u;
    for (int g = 0; g < NRANK; ++g) {
        int dst = rank;
        if (g != LOCALG) {
            const bool same_set = (SAMEM >> g) & 1u;
            int skip = __builtin_popcount((same_set ? SAMEM : ~SAMEM & ~(1u << LOCALG)) & ((1u << g) - 1u));
            for (int i = 1; i < NRANK; ++i) {
                const int p = (rank + i) & 7;
                if ((((CLQ >> p) ^ (CLQ >> rank) ^ 1u) & 1u) == (unsigned)same_set && skip-- == 0) { dst = p; break; }
            }
        }
        GroupParams& G = P.g[g];
        for (int v = 0; v < 8; ++v) {
            const unsigned phys = ((0x26543710u >> (4 * v)) + wsh) & 7u;
            G.wb[v] = w + dst * (NB * 4) + phys * 128;
            G.ptab |= phys << (4 * v);
        }
        G.out = (char*)d_out_all[dst] + (size_t)rank * ML * NB * 4;
        G.flag = (unsigned*)d_ws_all[dst] + WS_FLAG / 4 + rank * WG_PER_DST;
        G.x = (const float*)d_in[0];
        G.ws = (unsigned*)d_ws_all[rank];
        G.rank = (unsigned)rank;
        G.lout = (char*)d_out_all[rank];
        G.peer = (unsigned)dst;
        G.dbg = (unsigned long long*)((char*)d_ws_all[rank] + 65536);
    }
    HIP_CHECK(hipFuncSetAttribute((const void*)gemm_a2a_kernel, hipFuncAttributeMaxDynamicSharedMemorySize, LDS_BYTES));
    hipLaunchKernelGGL(gemm_a2a_kernel, dim3(NWG), dim3(NTHREADS), LDS_BYTES, stream, P);
    HIP_CHECK(hipGetLastError());
}
```

```cpp
#include <hip/hip_runtime.h>
#include <rccl/rccl.h>
#include <cstdio>
#include <cstdint>

#define HIP_CHECK(expr) do { hipError_t _e = (expr); if (_e != hipSuccess) { \
    fprintf(stderr, "HIP error %s at %s:%d\n", hipGetErrorString(_e), __FILE__, __LINE__); \
    return; } } while (0)

typedef _Float16 f16x8 __attribute__((ext_vector_type(8)));
typedef _Float16 f16x2 __attribute__((ext_vector_type(2)));
typedef float f32x4 __attribute__((ext_vector_type(4)));
typedef float f32x2 __attribute__((ext_vector_type(2)));
typedef unsigned u32x4 __attribute__((ext_vector_type(4)));
typedef unsigned u32x2 __attribute__((ext_vector_type(2)));
typedef unsigned u32x8 __attribute__((ext_vector_type(8)));
typedef __attribute__((address_space(3))) volatile unsigned lds_vu32;

constexpr int NRANK = 8;
constexpr int ML = 256;
constexpr int KD = 2048;
constexpr int ND = 2048;
constexpr int NB = 256;
constexpr int NWG = 256;
constexpr int WG_PER_DST = NWG / NRANK;
constexpr int NTHREADS = 640;

constexpr unsigned WS_SEQ = 0;
constexpr unsigned WS_FLAG = 4096;
constexpr unsigned WS_PROBE = 5120;
constexpr unsigned FLAG_COPY = 8192;
constexpr unsigned WS_TMO = 8192;
constexpr unsigned WS_X16 = 1u << 20;
constexpr unsigned WS_BYTES_USED = 3u << 20;
constexpr unsigned long long DEADLINE_TICKS = 50000000ull;
constexpr unsigned long long SELF_SERVE_TICKS = 3000ull;

#define LOCALG 4
struct __attribute__((aligned(128))) GroupParams {
    const char* wb[8];
    char* out;
    unsigned* flag;
    const float* x;
    unsigned* ws;
    unsigned ptab;
    unsigned rank;
    unsigned long long* dbg;
    char* lout;
    unsigned peer;
    unsigned pad;
};
struct Params { GroupParams g[NRANK]; };

constexpr int AUX_SYS = 17;
constexpr int AUX_DEV = 16;
constexpr int AUX_NT = 2;

__device__ __forceinline__ unsigned cvt2(float a, float b) {
    f32x2 v = {a, b};
    return __builtin_bit_cast(unsigned, __builtin_convertvector(v, f16x2));
}
__device__ __forceinline__ unsigned umin3(unsigned a, unsigned b, unsigned c) {
    return min(min(a, b), c);
}
#define SB() __builtin_amdgcn_sched_barrier(0)
#define BARRIER() do { SB(); asm volatile("s_barrier" ::: "memory"); SB(); } while (0)
#define STAMP(i)
#define PME 0x77777777u
#define PMO 0x11117777u
#define PML 4u
#define HT 0x012233u
#define NH 6
#define NFULL 4
#define CVO 1
#define DEP 4
#define WROLL 1
#define OFFT 0x76543210u
#define SXT 0x00000000u
#define SLOOP 0
#define LEXP 0
#define ONEBAR 1
#define PIDX 23
#define NPR 4
#define CLQ 0xB2u
#define SAMEM 0x0du
#define CW 1
#define CWLEN 13568
#define WDEP DEP

constexpr int LDS_RED = 4 * 8 * 2048;
constexpr int LDS_PAR = LDS_RED;
constexpr int LDS_CNT = LDS_RED + 64;
constexpr int LDS_BYTES = LDS_RED + 512;

__global__ void __launch_bounds__(NTHREADS) gemm_a2a_kernel(Params P) {
    extern __shared__ __attribute__((aligned(16))) char smem[];
    const int tid = threadIdx.x;
    const int lane = tid & 63;
    const int wave = __builtin_amdgcn_readfirstlane(tid >> 6) - 2;
    const int b = blockIdx.x;
    const int grp = b & 7, idx = b >> 3;
    const int mt = idx & 3, nv = idx >> 2;
    const int m0 = mt * 64;
    const unsigned sx = (SXT >> (4 * nv)) & 3u;
    STAMP(0);
    const char* ka = (const char*)__builtin_amdgcn_kernarg_segment_ptr() + grp * 128;
    unsigned long long pc;
    asm volatile("s_load_dword s100, %1, 0x0\n\ts_load_dword s101, %1, 0x40\n\ts_getpc_b64 %0" : "=s"(pc) : "s"(ka) : "s100", "s101", "memory");
    SB();
#define PTR(lo, hi) ((void*)(((unsigned long long)(hi) << 32) | (lo)))
#define RSRC(lo, hi, bytes) __builtin_amdgcn_make_buffer_rsrc(PTR(lo, hi), 0, bytes, 0x00020000)
    constexpr unsigned W_SPAN = (KD - 1) * ND * 4 + 128;
    const unsigned cw = CW ? __builtin_amdgcn_raw_buffer_load_b32(RSRC((unsigned)pc & ~255u, (unsigned)(pc >> 32), CWLEN),
                                                                  tid < 4 ? tid * 128u + idx * 512u : 0x80000000u, 0, 0) : 0u;
    SB();

    int lane8 = lane >> 3, lane16 = lane * 16, wv2 = wave;
    asm volatile("" : "+v"(lane8), "+v"(lane16), "+s"(wv2));
    if (__builtin_expect(wave < 0, 0)) {
    const int pw = wave + 2;
    __builtin_amdgcn_s_setprio(3);
    u32x2 misc;
    u32x8 ptrs;
    asm volatile("s_load_dwordx8 %0, %2, 0x40\n\ts_load_dwordx2 %1, %2, 0x60\n\ts_waitcnt lgkmcnt(0)" : "=&s"(ptrs), "=&s"(misc) : "s"(ka) : "memory");
    const __amdgpu_buffer_rsrc_t sr = RSRC(ptrs[6], ptrs[7], WS_BYTES_USED);
    const unsigned callv = __builtin_amdgcn_raw_buffer_load_b32(sr, 0, WS_SEQ + b * 4, AUX_DEV);
    if (!ONEBAR && (nv >= 4 || grp == LOCALG)) BARRIER();
    STAMP(10);
    const unsigned call = __builtin_amdgcn_readfirstlane(callv) + 1u;
    if (pw == 0 && lane == 0) *(unsigned*)(smem + LDS_PAR) = call & 1u;
    if (pw == 1 && lane < 8) *(unsigned*)(smem + LDS_CNT + lane * 4) = 0u;
    asm volatile("s_waitcnt lgkmcnt(0)" ::: "memory");
    if (ONEBAR && SLOOP) BARRIER();
    STAMP(1);
    const unsigned nt = (misc[0] >> (4 * nv)) & 7u;
    const __amdgpu_buffer_rsrc_t orr = __builtin_amdgcn_make_buffer_rsrc(PTR(ptrs[0], ptrs[1]), 0, ML * NB * 4, 0x00020000);
    const __amdgpu_buffer_rsrc_t fr = __builtin_amdgcn_make_buffer_rsrc(PTR(ptrs[2], ptrs[3]), 0, FLAG_COPY + WG_PER_DST * 4, 0x00020000);
    const unsigned ooff = (unsigned)((m0 + pw * 8 + (lane >> 3)) * NB + nt * 32 + (lane & 7) * 4) * 4u;
    const unsigned pm = grp == LOCALG ? PML : ((mt & 1 ? PMO : PME) >> (4 * nv)) & 7u;
    asm volatile("" :: "v"(ooff), "s"(pm));
    if (CW) asm volatile("" :: "v"(cw));
#pragma unroll
    for (int g = 0; g < (WROLL ? 1 : 7); ++g) BARRIER();
    const bool poller = pw == 1 && idx == PIDX && grp != LOCALG;
    unsigned fsel = 0;
    if (poller) {
        unsigned best[2] = {~0u, ~0u};
#pragma unroll 1
        for (int i = 0; i < NPR; ++i) {
#pragma unroll
            for (int c = 0; c < 2; ++c) {
                SB();
                const unsigned t0 = (unsigned)__builtin_amdgcn_s_memtime();
                asm volatile("s_waitcnt lgkmcnt(0)" ::: "memory");
                if (lane == 0) __builtin_amdgcn_raw_buffer_store_b32(call, sr, 0, WS_PROBE + c * FLAG_COPY + grp * 128, AUX_SYS);
                asm volatile("s_waitcnt vmcnt(0)" ::: "memory");
                const unsigned t1 = (unsigned)__builtin_amdgcn_s_memtime();
                SB();
                best[c] = min(best[c], t1 - t0);
            }
        }
        fsel = best[1] < best[0] ? FLAG_COPY : 0u;
    }
    bool timed_out = false;
    f32x4 sv[4];
#pragma unroll
    for (int st = 0; st < 4; ++st) {
        for (unsigned n = 1, t0 = 0;; ++n) {
            const unsigned c = *(lds_vu32*)(smem + LDS_CNT + (lane & 7) * 4);
            if (__all(c > (unsigned)st)) break;
            if (__builtin_expect((n & 4095u) == 0u, 0)) {
                const unsigned t = (unsigned)__builtin_amdgcn_s_memrealtime();
                if (n == 4096u) t0 = t;
                if (t - t0 > (unsigned)DEADLINE_TICKS) { timed_out = true; break; }
            }
        }
        asm volatile("" ::: "memory");
        SB();
        const char* red = smem + st * 16384 + pw * 1024 + lane * 16;
        f32x4 v[8];
#pragma unroll
        for (int w = 0; w < 8; ++w) v[w] = *(const f32x4*)(red + w * 2048);
        SB();
        f32x4 s = ((v[0] + v[1]) + (v[2] + v[3])) + ((v[4] + v[5]) + (v[6] + v[7]));
        s[0] = fmaxf(s[0], 0.f); s[1] = fmaxf(s[1], 0.f); s[2] = fmaxf(s[2], 0.f); s[3] = fmaxf(s[3], 0.f);
        sv[st] = s;
        if (__builtin_expect(timed_out, 0)) sv[st][0] = __builtin_nanf("");
#define PUSH(j) __builtin_amdgcn_raw_buffer_store_b128(__builtin_bit_cast(u32x4, sv[j]), orr, ooff, ((j) ^ sx) * (16 * NB * 4), AUX_SYS)
        if (st == 0) { if (pm & 1u) PUSH(0); }
        if (st == 1) { if (pm & 2u) { if (!(pm & 1u)) PUSH(0); PUSH(1); } }
        if (st == 2) { if (pm & 4u) { if (!(pm & 3u)) PUSH(0); if (!(pm & 2u)) PUSH(1); PUSH(2); } }
        if (st == 3) { if (!pm) PUSH(0); if (!(pm & 6u)) PUSH(1); if (!(pm & 4u)) PUSH(2); PUSH(3); }
        STAMP(3 + st);
    }
    {
        const u32x4 z = {0u, 0u, 0u, 0u};
        const unsigned so = WS_X16 + (((call & 1u) ^ 1u) << 20) + (unsigned)(b * 2 + pw) * 2048u;
        __builtin_amdgcn_raw_buffer_store_b128(z, sr, lane * 16, so, AUX_DEV);
        __builtin_amdgcn_raw_buffer_store_b128(z, sr, lane * 16 + 1024, so, AUX_DEV);
        asm volatile("s_waitcnt vmcnt(2)\n\ts_barrier" ::: "memory");
    }
    STAMP(7);
    if (__builtin_expect(timed_out, 0) && lane == 0) __builtin_amdgcn_raw_buffer_store_b32(call, sr, 0, WS_TMO, AUX_DEV);
    if (pw == 0) {
        if (lane < 2 && grp != LOCALG) __builtin_amdgcn_raw_buffer_store_b32(call, fr, lane * FLAG_COPY, idx * 4, AUX_SYS);
        if (lane == 0) __builtin_amdgcn_raw_buffer_store_b32(call, sr, 0, WS_SEQ + b * 4, AUX_DEV);
    }
    if (poller) {
        u32x4 lp;
        asm volatile("s_load_dwordx4 %0, %1, 0x70\n\ts_waitcnt lgkmcnt(0)" : "=&s"(lp) : "s"(ka) : "memory");
        const unsigned peer = lp[2] & 7u;
        const unsigned long long tw = __builtin_amdgcn_s_memrealtime();
        for (;;) {
            asm volatile("" ::: "memory");
            const unsigned f = __builtin_amdgcn_raw_buffer_load_b32(sr, (lane & 31) * 4, WS_FLAG + fsel + peer * 128u, AUX_SYS);
            if (__all(f >= call)) break;
            if (__builtin_amdgcn_s_memrealtime() - tw > DEADLINE_TICKS) { timed_out = true; break; }
        }
        if (timed_out && lane == 0) {
            __builtin_amdgcn_raw_buffer_store_b32(call, sr, 0, WS_TMO, AUX_DEV);
            __builtin_amdgcn_raw_buffer_store_b32(0x7fc00000u, RSRC(lp[0], lp[1], NRANK * ML * NB * 4), 0, peer * (ML * NB * 4), AUX_SYS);
        }
    }
    asm volatile("s_endpgm" ::: "memory");
    __builtin_amdgcn_s_waitcnt(0);
    }

    {
        if (__builtin_expect(grp == LOCALG, LEXP)) {
#define CLOAD(u, rb) _Pragma("unroll") for (int i = 0; i < 4; ++i) \
            px[4 * (u) + i] = __builtin_bit_cast(f32x4, __builtin_amdgcn_raw_buffer_load_b128(xr, cvo + (i >> 1) * 128, ((i & 1) * 8 + (rb) * 16) * KD * 4, AUX_NT))
#define CONVERT(u, rb) do { \
            const unsigned so = WS_X16 + (((__builtin_amdgcn_readfirstlane(callv) + 1u) & 1u) << 20) + (unsigned)((crb + (rb)) * 64 + cks) * 1024u; \
            const int c = lane & 7; \
            char* tb = smem + wave * 4096 + (u) * 2048; \
            _Pragma("unroll") for (int i = 0; i < 4; ++i) { \
                u32x2 h = {cvt2(px[4 * (u) + i][0], px[4 * (u) + i][1]), cvt2(px[4 * (u) + i][2], px[4 * (u) + i][3])}; \
                *(u32x2*)(tb + (i >> 1) * 1024 + ((i & 1) * 8 + (lane >> 3) + 16 * (c >> 1)) * 16 + (c & 1) * 8) = h; \
            } \
            asm volatile("s_waitcnt lgkmcnt(0)" ::: "memory"); \
            _Pragma("unroll") for (int j = 0; j < 2; ++j) { \
                u32x4 h = *(const u32x4*)(tb + j * 1024 + lane * 16); \
                _Pragma("unroll") for (int i = 0; i < 4; ++i) h[i] = h[i] == 0u ? 0x8000u : h[i]; \
                asm volatile("buffer_store_dwordx4 %0, %1, %2, %3 offen sc1\n\ts_nop 1" :: "v"(h), "v"(lane * 16 + j * 1024), "s"(sr), "s"(so) : "memory"); \
            } \
            SB(); } while (0)
            u32x4 xw;
            asm volatile("s_load_dwordx4 %0, %1, 0x50" : "=&s"(xw) : "s"(ka) : "memory");
            const int crb = (idx >> 3) * 4 + (wave >> 2), cks = ((idx & 7) * 4 + (wave & 3)) * 2;
            const unsigned cvo = (unsigned)((crb * 16 + (lane >> 3)) * KD + cks * 32 + (lane & 7) * 4) * 4u;
            asm volatile("s_waitcnt lgkmcnt(0)" : "+s"(xw) : "v"(cvo) : "memory");
            const __amdgpu_buffer_rsrc_t xr = RSRC(xw[0], xw[1], ML * KD * 4), sr = RSRC(xw[2], xw[3], WS_BYTES_USED);
            f32x4 px[8];
            const unsigned callv = __builtin_amdgcn_raw_buffer_load_b32(sr, 0, WS_SEQ + b * 4, AUX_DEV);
            CLOAD(0, 0);
            SB();
            CONVERT(0, 0);
            CLOAD(1, 2);
            SB();
            STAMP(13);
            CONVERT(1, 2);
            STAMP(14);
        }
        u32x2 wown, whelp;
        u32x4 xw;
        const int ht = (HT >> (4 * wave)) & 3;
        asm volatile("s_load_dwordx2 %0, %3, %4\n\ts_load_dwordx2 %1, %3, %5\n\ts_load_dwordx4 %2, %3, 0x50"
                     : "=&s"(wown), "=&s"(whelp), "=&s"(xw) : "s"(ka), "s"(nv * 8), "s"(ht * 8) : "memory");
        const int kw = wave * 256;
        const int rot = (2 * mt + (OFFT >> (4 * nv))) & 7;
        const unsigned woff = (unsigned)((kw + 4 * (lane >> 3)) * ND + 4 * (lane & 7)) * 4u;
        const unsigned wso0 = (unsigned)rot * (32 * ND * 4);
        const unsigned hvo = (unsigned)(((lane >> 3) * 256 + (lane & 7)) * ND) * 4u;
        const unsigned hr0 = (idx & 15) * 16 + (wave & 1) * 8, hr1 = ((2 * nv + (OFFT >> (4 * ht)) + 1) & 7) * 32 + mt * 8;
        unsigned hso = (wave < NFULL ? hr0 : hr1) * (ND * 4u);
        asm volatile("s_waitcnt lgkmcnt(0)" : "+s"(wown), "+s"(whelp), "+s"(xw) : "v"(woff), "s"(wso0), "v"(hvo), "s"(hso), "s"(hr0), "s"(hr1) : "memory");
        const __amdgpu_buffer_rsrc_t wr = RSRC(wown[0], wown[1], W_SPAN);
        unsigned pf = 0;
        if (nv >= 4 || grp == LOCALG) {
            if (nv >= 4 && wave < NH) pf = __builtin_amdgcn_raw_buffer_load_b32(RSRC(whelp[0], whelp[1], W_SPAN), hvo, hso, 0);
            BARRIER();
        }
        const __amdgpu_buffer_rsrc_t xr = RSRC(xw[0], xw[1], ML * KD * 4), sr = RSRC(xw[2], xw[3], WS_BYTES_USED);
        const int li = lane & 15, lg = lane >> 4;
        STAMP(25);

        f32x4 wv[8][4];
        u32x4 wf[8][2];
        u32x4 xf[32];
#define LDW1(i) wv[(i) >> 2][(i) & 3] = __builtin_bit_cast(f32x4, __builtin_amdgcn_raw_buffer_load_b128(wr, woff, (((((i) >> 2) + rot) & 7) * 32 + ((i) & 3)) * ND * 4, 0))
#define LDWG(s) do { LDW1(4 * (s)); LDW1(4 * (s) + 1); LDW1(4 * (s) + 2); LDW1(4 * (s) + 3); } while (0)
#define CVW(s) _Pragma("unroll") for (int t = 0; t < 2; ++t) _Pragma("unroll") for (int d = 0; d < 2; ++d) { \
        unsigned p0 = cvt2(wv[s][2 * d][t], wv[s][2 * d + 1][t]); \
        unsigned p2 = cvt2(wv[s][2 * d][2 + t], wv[s][2 * d + 1][2 + t]); \
        wf[s][t][d] = __builtin_amdgcn_update_dpp(p0, p2, 0x128, 0xf, 0xc, false); \
        wf[s][t][2 + d] = __builtin_amdgcn_update_dpp(p2, p0, 0x128, 0xf, 0x3, false); }
#define XLD(q) __builtin_amdgcn_raw_buffer_load_b128(sr, hoff, (hbase ^ (((q) >> 3) * 65536)) + ((((q) & 7) + rot) & 7) * 1024, AUX_DEV)
#define LDX1(q) xf[q] = XLD(q)
#define LDX4(q) do { LDX1(q); LDX1((q) + 1); LDX1((q) + 2); LDX1((q) + 3); } while (0)
#define WAITV(v) do { SB(); asm volatile("" :: "v"(v)); SB(); } while (0)
#define WAITW(s) WAITV(wv[s][3])
        const unsigned hoff = (unsigned)(lane * 16 + wave * 8192);
#pragma unroll
        for (int i = 0; i < WDEP; ++i) { LDW1(i); SB(); }
        if (!ONEBAR || !(nv >= 4 || grp == LOCALG)) BARRIER();
        const unsigned par = __builtin_amdgcn_readfirstlane(*(const unsigned*)(smem + LDS_PAR));
        const unsigned hbase = WS_X16 + (par << 20) + (unsigned)mt * 262144u + sx * 65536u;
        asm volatile("" :: "v"(pf), "s"(hbase));
        if (CW) asm volatile("" :: "v"(cw));
#pragma unroll
        for (int i = 0; i < 32; ++i) {
            WAITV(wv[i >> 2][i & 3]);
            if (i + WDEP < 32) LDW1((i + WDEP) & 31); else LDX1((i + WDEP) & 31);
            SB();
            if ((i & 3) == 3) { STAMP(17 + (i >> 2)); CVW(i >> 2); }
        }
        SB();
        constexpr int XPRE = WDEP;
        STAMP(2);

#define CNT_RETRY
#define MMA(f, s) do { \
            f16x8 af = __builtin_bit_cast(f16x8, f); \
            acc0 = __builtin_amdgcn_mfma_f32_16x16x32_f16(af, __builtin_bit_cast(f16x8, wf[s][0]), acc0, 0, 0, 0); \
            acc1 = __builtin_amdgcn_mfma_f32_16x16x32_f16(af, __builtin_bit_cast(f16x8, wf[s][1]), acc1, 0, 0, 0); } while (0)
#pragma unroll
        for (int st = 0; st < 4; ++st) {
            f32x4 acc0 = {0.f, 0.f, 0.f, 0.f}, acc1 = {0.f, 0.f, 0.f, 0.f};
            unsigned m = 0xffffffffu;
#pragma unroll
            for (int s = 0; s < 8; ++s) {
                const int q = st * 8 + s;
                WAITV(xf[q]);
                if (q + DEP >= XPRE && q + DEP < 32) LDX1((q + DEP) & 31);
                if (q == 0) { _Pragma("unroll") for (int e = XPRE; e < DEP; ++e) LDX1(e); }
                SB();
                MMA(xf[q], s);
                m = umin3(m, xf[q][0], xf[q][1]); m = umin3(m, xf[q][2], xf[q][3]);
            }
            if (st == 0) STAMP(12);
            if (__builtin_expect(__any(m == 0u), 0)) {
                const unsigned long long tw = __builtin_amdgcn_s_memrealtime();
                u32x4 f[8];
                for (;;) {
                    CNT_RETRY;
                    asm volatile("" ::: "memory");
                    m = 0xffffffffu;
#pragma unroll
                    for (int s = 0; s < 8; ++s) f[s] = XLD(st * 8 + s);
#pragma unroll
                    for (int s = 0; s < 8; ++s) { m = umin3(m, f[s][0], f[s][1]); m = umin3(m, f[s][2], f[s][3]); }
                    if (!__any(m == 0u)) break;
                    if (__builtin_amdgcn_s_memrealtime() - tw > SELF_SERVE_TICKS) {
                        const unsigned fo = (unsigned)((m0 + (st ^ sx) * 16 + li) * KD + kw + lg * 8) * 4u;
#pragma unroll
                        for (int s = 0; s < 8; ++s) {
                            f32x4 qa = __builtin_bit_cast(f32x4, __builtin_amdgcn_raw_buffer_load_b128(xr, fo, ((s + rot) & 7) * 128, 0));
                            f32x4 qb = __builtin_bit_cast(f32x4, __builtin_amdgcn_raw_buffer_load_b128(xr, fo + 16, ((s + rot) & 7) * 128, 0));
                            f[s] = u32x4{cvt2(qa[0], qa[1]), cvt2(qa[2], qa[3]), cvt2(qb[0], qb[1]), cvt2(qb[2], qb[3])};
                            SB();
                        }
                        break;
                    }
                }
                acc0 = f32x4{0.f, 0.f, 0.f, 0.f}; acc1 = f32x4{0.f, 0.f, 0.f, 0.f};
#pragma unroll
                for (int s = 0; s < 8; ++s) MMA(f[s], s);
            }
            char* red = smem + st * 16384;
#pragma unroll
            for (int r = 0; r < 4; ++r) {
                f32x2 v = {acc0[r], acc1[r]};
                *(f32x2*)(red + wave * 2048 + (lg * 4 + r) * 128 + (lane & 7) * 16 + ((lane >> 3) & 1) * 8) = v;
            }
            SB();
            asm volatile("" ::: "memory");
            *(lds_vu32*)(smem + LDS_CNT + wave * 4) = st + 1;
            SB();
            if (st == 0) STAMP(11);
        }
    }
}

extern "C" void kernel_launch(int rank, int nranks, ncclComm_t comm,
                              hipStream_t stream,
                              void* const* d_in, const int* in_sizes, int n_in,
                              void* d_out, int out_size,
                              const int* in_dtypes, int out_dtype,
                              void* const* d_out_all,
                              void* const* d_ws_all, size_t ws_bytes) {
    if (nranks != NRANK || (unsigned)rank >= (unsigned)NRANK || ws_bytes < WS_BYTES_USED) {
        fprintf(stderr, "unexpected configuration\n");
        return;
    }
    Params P = {};
    const char* w = (const char*)d_in[1];
    const unsigned wsh = (0u - (unsigned)((uintptr_t)w >> 7)) & 7u;
    for (int g = 0; g < NRANK; ++g) {
        int dst = rank;
        if (g != LOCALG) {
            const bool same_set = (SAMEM >> g) & 1u;
            int skip = __builtin_popcount((same_set ? SAMEM : ~SAMEM & ~(1u << LOCALG)) & ((1u << g) - 1u));
            for (int i = 1; i < NRANK; ++i) {
                const int p = (rank + i) & 7;
                if ((((CLQ >> p) ^ (CLQ >> rank) ^ 1u) & 1u) == (unsigned)same_set && skip-- == 0) { dst = p; break; }
            }
        }
        GroupParams& G = P.g[g];
        for (int v = 0; v < 8; ++v) {
            const unsigned phys = ((0x26543710u >> (4 * v)) + wsh) & 7u;
            G.wb[v] = w + dst * (NB * 4) + phys * 128;
            G.ptab |= phys << (4 * v);
        }
        G.out = (char*)d_out_all[dst] + (size_t)rank * ML * NB * 4;
        G.flag = (unsigned*)d_ws_all[dst] + WS_FLAG / 4 + rank * WG_PER_DST;
        G.x = (const float*)d_in[0];
        G.ws = (unsigned*)d_ws_all[rank];
        G.rank = (unsigned)rank;
        G.lout = (char*)d_out_all[rank];
        G.peer = (unsigned)dst;
        G.dbg = (unsigned long long*)((char*)d_ws_all[rank] + 65536);
    }
    HIP_CHECK(hipFuncSetAttribute((const void*)gemm_a2a_kernel, hipFuncAttributeMaxDynamicSharedMemorySize, LDS_BYTES));
    hipLaunchKernelGGL(gemm_a2a_kernel, dim3(NWG), dim3(NTHREADS), LDS_BYTES, stream, P);
    HIP_CHECK(hipGetLastError());
}
```
